```python
import numpy as np

kernel_name = "gpu_ag_gemm_m4096_k4096_n4096_bf16_reluq8"


ABSMAX_RTOL = 0.02

COMPUTE_DTYPE = "bfloat16"
EPILOGUE = "relu_quant_int8"
IN_QUANT = None
OUT_QUANT = "int8"
IN_DTYPES = {"x": "bfloat16", "w_mat": "bfloat16"}
WORLD_SIZE = 8

def _to_bf16(x):
    u = np.ascontiguousarray(x, dtype=np.float32).view(np.uint32)
    return ((u + 0x7FFF + ((u >> 16) & 1)) & 0xFFFF0000).view(np.float32)

def setup_inputs():

    x = _to_bf16(np.random.randn(4096, 4096).astype(np.float32))
    w_mat = _to_bf16((np.random.randn(4096, 4096) / np.sqrt(4096)).astype(np.float32))
    return {"x": x, "w_mat": w_mat}

def _quant_dequant(y, amax):
    scale = amax / 127.0
    q = np.clip(np.round(y / scale), -127.0, 127.0)
    return (q * scale).astype(np.float32)
def _epilogue(y):
    y = np.maximum(y, 0.0)
    return _quant_dequant(y, np.abs(y).max())

def reference(x, w_mat):
    y = (x @ w_mat).astype(np.float32)
    return _epilogue(y)
```

```cpp
#include <hip/hip_runtime.h>
#include <rccl/rccl.h>
#include <cstdio>
#include <cstdint>

#define LAS __attribute__((address_space(3)))
#define GAS __attribute__((address_space(1)))
typedef unsigned v4u __attribute__((ext_vector_type(4)));
typedef unsigned v2u __attribute__((ext_vector_type(2)));
typedef float f32x4 __attribute__((ext_vector_type(4)));
typedef __bf16 bf16x8 __attribute__((ext_vector_type(8)));
typedef unsigned long long u64;

#ifndef CU_PER_DEST
#define CU_PER_DEST 8
#endif
#define WV_PER_CU 3
#define DYN 2
#define POLL_SLEEP 0
#define DMA_AUX 17

constexpr int NR = 8;
constexpr int NWG = 256;
constexpr int NGW = 128;
constexpr int KSTEPS = 64;
constexpr int NSLOT = 4;
constexpr unsigned A_OFF = 0;
constexpr unsigned B_OFF = 16u << 20;
constexpr unsigned F_OFF = 24u << 20;
constexpr unsigned FA_OFF = F_OFF;
constexpr unsigned FB_OFF = F_OFF + 8192;
constexpr unsigned Q_OFF = F_OFF + 12288;
constexpr unsigned TM_OFF = F_OFF + 14336;
constexpr unsigned AM_OFF = F_OFF + 16384;
constexpr unsigned DN_OFF = F_OFF + 32768;
constexpr unsigned CT_OFF = F_OFF + 49152;
constexpr unsigned DBG_OFF = F_OFF + 65536;
constexpr unsigned WS_USED = F_OFF + 65536 + 131072;
constexpr unsigned X_BYTES = 512u * 4096u * 2u;
constexpr unsigned W_BYTES = 4096u * 512u * 2u;
constexpr unsigned OUT_BYTES = 4096u * 512u;
constexpr int SLOT_BYTES = 32768;
constexpr int RING_BYTES = NSLOT * SLOT_BYTES;
constexpr int EPI_OFF = RING_BYTES;
constexpr int CTL_OFF = EPI_OFF + 16384;
constexpr int LDS_BYTES = CTL_OFF + 128;
constexpr u64 DEADLINE_TICKS = 40000000ull;
constexpr int RSRC3 = 0x00020000;
static_assert(LDS_BYTES <= 160 * 1024, "LDS");

struct Params {
    const void* x;
    const void* w;
    char* ws[NR];
    char* out[NR];
    int rank;
};

__device__ __forceinline__ u64 rt() { return __builtin_amdgcn_s_memrealtime(); }
__device__ __forceinline__ __amdgpu_buffer_rsrc_t mkrsrc(const void* p, unsigned bytes) {
    return __builtin_amdgcn_make_buffer_rsrc((void*)p, (short)0, (int)bytes, RSRC3);
}
#define WAIT_VM0() asm volatile("s_waitcnt vmcnt(0)" ::: "memory")
#define WAIT_VM0_B() do { asm volatile("" ::: "memory"); __builtin_amdgcn_s_waitcnt(0x0F70); asm volatile("" ::: "memory"); } while (0)
#define WAIT_LGKM0() asm volatile("s_waitcnt lgkmcnt(0)" ::: "memory")

#define STAMP(idx) do { } while (0)

__device__ __forceinline__ unsigned wave_umax(unsigned v) {
    v = max(v, (unsigned)__builtin_amdgcn_update_dpp(0, (int)v, 0x111, 0xf, 0xf, true));
    v = max(v, (unsigned)__builtin_amdgcn_update_dpp(0, (int)v, 0x112, 0xf, 0xf, true));
    v = max(v, (unsigned)__builtin_amdgcn_update_dpp(0, (int)v, 0x114, 0xf, 0xf, true));
    v = max(v, (unsigned)__builtin_amdgcn_update_dpp(0, (int)v, 0x118, 0xf, 0xf, true));
    const unsigned a = __builtin_amdgcn_readlane(v, 15), b = __builtin_amdgcn_readlane(v, 31);
    const unsigned c = __builtin_amdgcn_readlane(v, 47), d = __builtin_amdgcn_readlane(v, 63);
    return max(max(a, b), max(c, d));
}

__global__ void __launch_bounds__(1024, 1) ag_gemm_kernel(Params p) {
    __shared__ __attribute__((aligned(16))) unsigned char lds[LDS_BYTES];
    LAS unsigned char* L8 = (LAS unsigned char*)lds;
    volatile LAS unsigned* CTL = (volatile LAS unsigned*)(L8 + CTL_OFF);
    const int tid = threadIdx.x;
    const int wave = __builtin_amdgcn_readfirstlane(tid >> 6);
    const int lane = tid & 63;
    const unsigned rank = (unsigned)p.rank & 7u;
    const unsigned grp = rank >> 2, mem = rank & 3u;
    const unsigned b = blockIdx.x;
    char* const wsme = p.ws[rank];
    GAS unsigned* ctr = (GAS unsigned*)(wsme + CT_OFF + (b * 16 + wave) * 4);

    if (b < (unsigned)(NWG - NGW)) {
        const unsigned dsel = (b >> 3) & 7u;
        const unsigned sub = (b & 7u) + 8u * (b >> 6);
        if (sub >= (unsigned)CU_PER_DEST || wave >= WV_PER_CU) return;
        constexpr unsigned NWV = CU_PER_DEST * WV_PER_CU;
        const unsigned wv = (unsigned)wave * CU_PER_DEST + sub;
        const unsigned dst = (rank + dsel) & 7u;
        const bool self = dsel == 0u;
        const bool same = (dst >> 2) == grp;
        const unsigned nper = self ? 13u : 5u;
        const unsigned nx = same ? 5u : 3u;
        const unsigned nunits = nper * KSTEPS;
        char* const wsd = p.ws[dst];
        const __amdgpu_buffer_rsrc_t rd = mkrsrc(wsd, WS_USED);
        const unsigned xoff = (unsigned)(lane >> 3) * 8192u + (unsigned)((lane & 7) ^ (lane >> 3)) * 16u;
        const unsigned q = lane >> 4, i = lane & 15;
        const unsigned woff = q * 8192u + i * 16u;
        const unsigned wst = (i >> 3) * 4096u + (q * 16u + (i & 7u)) * 16u;
        GAS unsigned* qhead = (GAS unsigned*)(wsme + Q_OFF + dsel * 128u);
        v4u cur[8];
        auto load_unit = [&](unsigned u) {
            const unsigned ks = u / nper, idx = u - ks * nper;
            const bool isx = idx < nx;
            const unsigned xb = same ? idx : 5u + idx;
            const unsigned wi = idx - nx;
            const unsigned j = self ? (wi >> 1) : (dst & 3u), kb = wi & 1u;
            const unsigned so = isx ? xb * (64u * 8192u) + ks * 128u : (ks * 64u + kb * 32u) * 1024u + j * 256u;
            const unsigned st = isx ? 65536u : 1024u;
            const unsigned vo = isx ? xoff : woff;
            const __amdgpu_buffer_rsrc_t rs = mkrsrc(isx ? p.x : p.w, X_BYTES);
#pragma unroll
            for (int t = 0; t < 8; ++t) cur[t] = __builtin_amdgcn_raw_buffer_load_b128(rs, vo, so + t * st, 0);
        };
        load_unit(wv);
        const unsigned call_v = __hip_atomic_load(ctr, __ATOMIC_RELAXED, __HIP_MEMORY_SCOPE_AGENT) + 1u;
        WAIT_VM0_B();
        const unsigned call = __builtin_amdgcn_readfirstlane(call_v);
        GAS unsigned* pend = ctr;
        const unsigned qbase = (call - 1u) * nunits - NWV;
        if (wave == 0) STAMP(0);
        const unsigned qbase2 = (call - 1u) * nunits - 2u * NWV;
        unsigned u = wv, un = wv + NWV;
        while (u < nunits) {
            unsigned tk = 0;
            const unsigned ks = u / nper, idx = u - ks * nper;
            const bool isx = idx < nx;
            const unsigned drb = same ? mem * 5u + idx : 20u + mem * 3u + idx;
            const unsigned wi = idx - nx;
            const unsigned dcb = self ? (wi >> 1) : 4u + mem, kb = wi & 1u;
            v4u o[8];
            if (isx) {
#pragma unroll
                for (int t = 0; t < 8; ++t) o[t] = cur[t];
            } else {
#pragma unroll
                for (int d = 0; d < 4; ++d) {
#pragma unroll
                    for (int e = 0; e < 4; ++e) {
                        o[2 * d][e] = __builtin_amdgcn_perm(cur[2 * e + 1][d], cur[2 * e][d], 0x05040100u);
                        o[2 * d + 1][e] = __builtin_amdgcn_perm(cur[2 * e + 1][d], cur[2 * e][d], 0x07060302u);
                    }
                }
            }
            asm volatile("s_mov_b64 exec, 1\n\t"
                         "global_store_dword %1, %4, %5 sc0 sc1\n\t"
                         "global_atomic_add %0, %1, %2, %3 sc0\n\t"
                         "s_mov_b64 exec, -1"
                         : "=&v"(tk) : "v"(0u), "v"(1u), "s"(qhead), "v"(call), "s"(pend) : "memory");
            load_unit(un < nunits ? un : u);
            asm volatile("" ::: "memory");
            const unsigned so = isx ? A_OFF + (ks * 32u + drb) * 8192u : B_OFF + (ks * 8u + dcb) * 16384u + kb * 8192u;
            const unsigned so2 = so + (isx ? 4096u : 128u);
            const unsigned vo = isx ? (unsigned)lane * 16u : wst;
            const unsigned fpos = (ks & 3u) * 128u + (ks >> 2) * 8u;
            const unsigned foff = isx ? FA_OFF + (drb >> 1) * 512u + fpos + (drb & 1u) * 4u : FB_OFF + dcb * 512u + fpos + kb * 4u;
#pragma unroll
            for (int t = 0; t < 8; ++t)
                __builtin_amdgcn_raw_buffer_store_b128(o[t], rd, vo + (t & 3) * 1024u, t < 4 ? so : so2, 17);
            WAIT_VM0_B();
            asm volatile("" : "+v"(tk) :: "memory");
            pend = (GAS unsigned*)(wsd + foff);
            u = un;
            un = __builtin_amdgcn_readfirstlane(tk) - qbase2;
        }
        if (lane == 0) __hip_atomic_store(pend, call, __ATOMIC_RELAXED, __HIP_MEMORY_SCOPE_SYSTEM);
        STAMP(16 + wave);
        return;
    }

    const u64 t0 = rt();
    const unsigned call_v = __hip_atomic_load(ctr, __ATOMIC_RELAXED, __HIP_MEMORY_SCOPE_AGENT) + 1u;
    if (tid < 32) CTL[tid] = 0u;
    const unsigned gb = b - (unsigned)(NWG - NGW);
    const unsigned xcd = gb & 7u, sl = gb >> 3;
    const unsigned rbp = (xcd >> 1) * 4u + (sl & 3u);
    const unsigned cbi = sl >> 2;
    const unsigned cb = (cbi & 1u) + 2u * (xcd & 1u) + 4u * (cbi >> 1);
    WAIT_LGKM0();
    __builtin_amdgcn_s_barrier();
    const unsigned call = __builtin_amdgcn_readfirstlane(call_v);
    if (lane == 0) __hip_atomic_store(ctr, call, __ATOMIC_RELAXED, __HIP_MEMORY_SCOPE_AGENT);
    GAS unsigned* tmo = (GAS unsigned*)(wsme + TM_OFF);

    if (wave >= 8) {
        __builtin_amdgcn_s_setprio(2);
        const unsigned s = (unsigned)wave & 3u, h = ((unsigned)wave >> 2) & 1u;
        const __amdgpu_buffer_rsrc_t rs = mkrsrc(wsme, WS_USED);
        const GAS u64* fp = (const GAS u64*)(wsme + (h ? FB_OFF + cb * 512u : FA_OFF + rbp * 512u) + s * 128u + (lane & 15) * 8u);
        const unsigned src0 = h ? B_OFF + (s * 8u + cb) * 16384u : A_OFF + (s * 32u + rbp * 2u) * 8192u;
        const unsigned sstep = h ? 4u * 8u * 16384u : 4u * 32u * 8192u;
        LAS unsigned char* dst = L8 + s * SLOT_BYTES + h * 16384u;
        unsigned known = 0;
        bool dead = false;
        if (wave == 8) STAMP(2);
        for (unsigned n = 0; n < 16u; ++n) {
            while (((known >> n) & 1u) == 0u) {
                const u64 f = __hip_atomic_load(fp, __ATOMIC_RELAXED, __HIP_MEMORY_SCOPE_SYSTEM);
                known = (unsigned)__ballot((unsigned)f >= call && (unsigned)(f >> 32) >= call);
                if (((known >> n) & 1u) == 0u) {
                    if (rt() - t0 > DEADLINE_TICKS) { dead = true; break; }
                    __builtin_amdgcn_s_sleep(POLL_SLEEP);
                }
            }
            while (!dead && CTL[12 + s] < 8u * n) {
                if (CTL[8] != 0u) dead = true;
                __builtin_amdgcn_s_sleep(1);
            }
            if (dead) break;
            asm volatile("" ::: "memory");
            const unsigned sa = src0 + n * sstep;
#pragma unroll
            for (int g = 0; g < 4; ++g) {
                __builtin_amdgcn_raw_ptr_buffer_load_lds(rs, dst + g * 4096, 16, lane * 16u, sa + g * 4096u, 0, DMA_AUX);
                __builtin_amdgcn_raw_ptr_buffer_load_lds(rs, dst + g * 4096, 16, lane * 16u, sa + g * 4096u, 1024, DMA_AUX);
                __builtin_amdgcn_raw_ptr_buffer_load_lds(rs, dst + g * 4096, 16, lane * 16u, sa + g * 4096u, 2048, DMA_AUX);
                __builtin_amdgcn_raw_ptr_buffer_load_lds(rs, dst + g * 4096, 16, lane * 16u, sa + g * 4096u, 3072, DMA_AUX);
            }
            WAIT_VM0();
            CTL[s * 2u + h] = n + 1u;
        }
        if (dead) {
            __hip_atomic_store(tmo, call, __ATOMIC_RELAXED, __HIP_MEMORY_SCOPE_AGENT);
            CTL[8] = 1u; CTL[9] = 1u;
        }
        STAMP(48 + (wave - 8));
        return;
    }

    const int wm = wave >> 1, wn = wave & 1;
    const int g = lane >> 4, i = lane & 15;
    char* const am_dst = p.ws[lane & 7] + AM_OFF + (rank * 128u + gb) * 8u;
    f32x4 acc[2][4];
#pragma unroll
    for (int m = 0; m < 2; ++m)
#pragma unroll
        for (int f = 0; f < 4; ++f) acc[m][f] = (f32x4){0.f, 0.f, 0.f, 0.f};
    const unsigned a_off0 = (unsigned)(32 * wm + i) * 128u + (unsigned)((g ^ (i & 7)) * 16);
    const unsigned a_off1 = (unsigned)(32 * wm + i) * 128u + (unsigned)(((4 + g) ^ (i & 7)) * 16);
    const unsigned b_off = 16384u + (unsigned)wn * 4096u + (unsigned)lane * 16u;
    bool dead = false;
    unsigned mycnt = 0;
    unsigned last = 3;
    const volatile LAS v2u* fullp = (const volatile LAS v2u*)(L8 + CTL_OFF + (lane < 4 ? lane : 4) * 8);
    for (unsigned n = 0; n < (unsigned)KSTEPS; ++n) {
        unsigned m;
        while (true) {
            const v2u f = *fullp;
            m = (unsigned)__ballot(min(f.x, f.y) > mycnt) & 31u;
            if (m) break;
            __builtin_amdgcn_s_sleep(1);
        }
        if (m & 16u) { dead = true; break; }
        const unsigned s = (last + 1u + (unsigned)__builtin_ctz((m | (m << 4)) >> (last + 1u))) & 3u;
        last = s;
        asm volatile("" ::: "memory");
        const LAS unsigned char* sp = L8 + s * SLOT_BYTES;
        v4u af[2][2], bf[2][4];
#pragma unroll
        for (int mm = 0; mm < 2; ++mm) {
            af[0][mm] = *(const LAS v4u*)(sp + a_off0 + mm * 2048);
            af[1][mm] = *(const LAS v4u*)(sp + a_off1 + mm * 2048);
        }
#pragma unroll
        for (int kb = 0; kb < 2; ++kb)
#pragma unroll
            for (int f = 0; f < 4; ++f) bf[kb][f] = *(const LAS v4u*)(sp + b_off + kb * 8192 + f * 1024);
        mycnt += ((unsigned)lane == s) ? 1u : 0u;
        WAIT_LGKM0();
        if (lane == 0) __hip_atomic_fetch_add((LAS unsigned*)(L8 + CTL_OFF + 48 + s * 4u), 1u, __ATOMIC_RELAXED, __HIP_MEMORY_SCOPE_WORKGROUP);
        asm volatile("" : "+v"(af[0][0]), "+v"(af[0][1]), "+v"(af[1][0]), "+v"(af[1][1]),
                          "+v"(bf[0][0]), "+v"(bf[0][1]), "+v"(bf[0][2]), "+v"(bf[0][3]),
                          "+v"(bf[1][0]), "+v"(bf[1][1]), "+v"(bf[1][2]), "+v"(bf[1][3]) :: "memory");
#pragma unroll
        for (int kb = 0; kb < 2; ++kb)
#pragma unroll
            for (int mm = 0; mm < 2; ++mm)
#pragma unroll
                for (int f = 0; f < 4; ++f)
                    acc[mm][f] = __builtin_amdgcn_mfma_f32_16x16x32_bf16(__builtin_bit_cast(bf16x8, af[kb][mm]),
                                                                         __builtin_bit_cast(bf16x8, bf[kb][f]), acc[mm][f], 0, 0, 0);
    }
    if (wave == 0) STAMP(5);

    {
        float mx = 0.f;
#pragma unroll
        for (int m = 0; m < 2; ++m)
#pragma unroll
            for (int f = 0; f < 4; ++f)
#pragma unroll
                for (int r = 0; r < 4; ++r) mx = fmaxf(mx, acc[m][f][r]);
        const unsigned wmx = wave_umax(__float_as_uint(mx));
        unsigned cnt, bits;
        const unsigned addr = (unsigned)(CTL_OFF + 64), one = 1u;
        asm volatile("s_mov_b64 exec, 1\n\t"
                     "ds_max_u32 %2, %3\n\t"
                     "ds_add_rtn_u32 %0, %2, %4 offset:4\n\t"
                     "s_mov_b64 exec, -1\n\t"
                     "ds_read_b32 %1, %2\n\t"
                     "s_waitcnt lgkmcnt(0)"
                     : "=&v"(cnt), "=&v"(bits) : "v"(addr), "v"(wmx), "v"(one) : "memory");
        cnt = __builtin_amdgcn_readfirstlane(cnt);
        if (cnt == 7u && lane < 8)
            __hip_atomic_store((GAS u64*)am_dst, ((u64)call << 32) | bits, __ATOMIC_RELAXED, __HIP_MEMORY_SCOPE_SYSTEM);
    }
    if (wave == 0) STAMP(6);
    float amax;
    {
        const __amdgpu_buffer_rsrc_t rs = mkrsrc(wsme, WS_USED);
        unsigned gm = 0;
        while (true) {
            const v4u v = __builtin_amdgcn_raw_buffer_load_b128(rs, lane * 16u, AM_OFF + wave * 1024u, 17);
            const unsigned tm = __hip_atomic_load(tmo, __ATOMIC_RELAXED, __HIP_MEMORY_SCOPE_AGENT);
            const bool ok = v.y >= call && v.w >= call;
            gm = max(v.x, v.z);
            if (__builtin_amdgcn_readfirstlane(tm) == call) dead = true;
            if (__all(ok) || dead) break;
            if (rt() - t0 > DEADLINE_TICKS) {
                __hip_atomic_store(tmo, call, __ATOMIC_RELAXED, __HIP_MEMORY_SCOPE_AGENT);
                dead = true;
                break;
            }
        }
        const unsigned gmx = dead ? 0xffffffffu : wave_umax(gm);
        const unsigned addr = (unsigned)(CTL_OFF + 72), one = 1u;
        asm volatile("s_mov_b64 exec, 1\n\t"
                     "ds_max_u32 %0, %1 offset:4\n\t"
                     "ds_add_u32 %0, %2\n\t"
                     "s_mov_b64 exec, -1"
                     :: "v"(addr), "v"(gmx), "v"(one) : "memory");
        v2u r;
        do { r = *(const volatile LAS v2u*)(L8 + CTL_OFF + 72); } while (r.x < 8u);
        dead = r.y == 0xffffffffu;
        amax = __uint_as_float(r.y);
    }
    if (wave == 0) STAMP(7);
    const float inv = amax > 0.f ? 127.0f / amax : 0.f;

    LAS unsigned char* ep = L8 + EPI_OFF + wave * 2048;
#pragma unroll
    for (int m = 0; m < 2; ++m)
#pragma unroll
        for (int r = 0; r < 4; ++r) {
            unsigned pk = 0;
#pragma unroll
            for (int f = 0; f < 4; ++f) pk = __builtin_amdgcn_cvt_pk_u8_f32(__builtin_rintf(acc[m][f][r] * inv), f, pk);
            if (dead) pk = 0x80808080u;
            *(LAS unsigned*)(ep + (16 * m + 4 * g + r) * 64 + 8 * (i & 7) + 4 * (i >> 3)) = pk;
        }
    WAIT_LGKM0();
    const unsigned rb = rbp * 2u + ((unsigned)wm >> 1);
    const unsigned srank = rb < 20u ? grp * 4u + rb / 5u : (grp ^ 1u) * 4u + (rb - 20u) / 3u;
    const unsigned sblk = rb < 20u ? rb % 5u : 5u + (rb - 20u) % 3u;
    const unsigned grow = srank * 512u + sblk * 64u + ((unsigned)wm & 1u) * 32u;
    const bool remote = cb >= 4u;
    const unsigned orank = remote ? (grp ^ 1u) * 4u + (cb - 4u) : rank;
    const unsigned ocol = (remote ? mem : cb) * 128u + (unsigned)wn * 64u;
    const __amdgpu_buffer_rsrc_t ro = mkrsrc(p.out[orank & 7u], OUT_BYTES);
    {
        const v4u o0 = *(const LAS v4u*)(ep + lane * 16);
        const v4u o1 = *(const LAS v4u*)(ep + 1024 + lane * 16);
        const unsigned vo = (unsigned)(lane >> 2) * 512u + (unsigned)(lane & 3) * 16u;
        __builtin_amdgcn_raw_buffer_store_b128(o0, ro, vo, grow * 512u + ocol, 17);
        __builtin_amdgcn_raw_buffer_store_b128(o1, ro, vo, (grow + 16u) * 512u + ocol, 17);
    }
    if (remote) {
        WAIT_VM0();
        if (lane == 0)
            __hip_atomic_store((GAS unsigned*)(p.ws[orank & 7u] + DN_OFF + (mem * 16u + rbp) * 128u + wave * 4u), call,
                               __ATOMIC_RELAXED, __HIP_MEMORY_SCOPE_SYSTEM);
    } else {
        const GAS unsigned* dn = (const GAS unsigned*)(wsme + DN_OFF + (cb * 16u + rbp) * 128u + wave * 4u);
        while (__hip_atomic_load(dn, __ATOMIC_RELAXED, __HIP_MEMORY_SCOPE_SYSTEM) < call) {
            if (dead || rt() - t0 > DEADLINE_TICKS) {
                __hip_atomic_store(tmo, call, __ATOMIC_RELAXED, __HIP_MEMORY_SCOPE_AGENT);
                WAIT_VM0();
                if (lane == 0) __builtin_amdgcn_raw_buffer_store_b8((unsigned char)0x80, ro, 0u, grow * 512u + ocol, 17);
                break;
            }
        }
    }
    if (wave == 0) STAMP(8);
}

extern "C" void kernel_launch(int rank, int nranks, ncclComm_t comm, hipStream_t stream,
                              void* const* d_in, const int* in_sizes, int n_in,
                              void* d_out, int out_size, const int* in_dtypes, int out_dtype,
                              void* const* d_out_all, void* const* d_ws_all, size_t ws_bytes) {
    if (nranks != NR || n_in < 2 || ws_bytes < WS_USED || (unsigned)rank >= (unsigned)NR ||
        in_sizes[0] != 512 * 4096 || in_sizes[1] != 4096 * 512 || out_size != 4096 * 512) {
        fprintf(stderr, "unexpected problem configuration\n");
        return;
    }
    Params p;
    p.x = d_in[0];
    p.w = d_in[1];
    for (int r = 0; r < NR; ++r) { p.ws[r] = (char*)d_ws_all[r]; p.out[r] = (char*)d_out_all[r]; }
    p.rank = rank;
    ag_gemm_kernel<<<dim3(NWG), dim3(1024), 0, stream>>>(p);
}
```

```cpp
#include <hip/hip_runtime.h>
#include <rccl/rccl.h>
#include <cstdio>
#include <cstdint>

#define LAS __attribute__((address_space(3)))
#define GAS __attribute__((address_space(1)))
typedef unsigned v4u __attribute__((ext_vector_type(4)));
typedef unsigned v2u __attribute__((ext_vector_type(2)));
typedef float f32x4 __attribute__((ext_vector_type(4)));
typedef float f32x2 __attribute__((ext_vector_type(2)));
typedef __bf16 bf16x8 __attribute__((ext_vector_type(8)));
typedef unsigned long long u64;

#ifndef CU_PER_DEST
#define CU_PER_DEST 8
#endif
#define WV_PER_CU 3
#define WIN 6
#define SELF_CU 16
#define SELF_WV 4
#define OUT_WIN 4
#define XCOPY 1
#define WLIN 1
#define FASTG 1
#define COVL 1
#define WARM 1
#define AP 4
#define CSLEEP 1
#define LSLEEP 1
#define OWNROWS 1

constexpr int NR = 8;
constexpr int NWG = 256;
constexpr int NGW = 128;
constexpr int KSTEPS = 64;
constexpr int NSLOT = 4;
constexpr unsigned NRB = 36;
constexpr unsigned A_OFF = 0;
constexpr unsigned B_OFF = 18u << 20;
constexpr unsigned F_OFF = 26u << 20;
constexpr unsigned FA_OFF = F_OFF;
constexpr unsigned FB_OFF = F_OFF + 12288;
constexpr unsigned Q_OFF = F_OFF + 16384;
constexpr unsigned TM_OFF = F_OFF + 18432;
constexpr unsigned G1_OFF = F_OFF + 24576;
constexpr unsigned CT_OFF = F_OFF + 32768;
constexpr unsigned G0_OFF = F_OFF + 49152;
constexpr unsigned G2_OFF = F_OFF + 65536;
constexpr unsigned DBG_OFF = F_OFF + 131072;
constexpr unsigned WS_USED = F_OFF + 131072 + 131072;
constexpr unsigned X_BYTES = 512u * 4096u * 2u;
constexpr unsigned OUT_BYTES = 4096u * 512u;
constexpr int SLOT_BYTES = 32768;
constexpr int CTL_OFF = NSLOT * SLOT_BYTES;
constexpr int LDS_BYTES = CTL_OFF + 128;
constexpr u64 DEADLINE_TICKS = 40000000ull;
constexpr int RSRC3 = 0x00020000;
constexpr unsigned BAD = 0xffffffffu;
static_assert(LDS_BYTES <= 160 * 1024, "LDS");

struct Params {
    const void* x;
    const void* w;
    char* wsr[NR];
    char* outr[NR];
    int rank;
};

__device__ __forceinline__ unsigned opaque(unsigned v) { asm volatile("" : "+s"(v)); return v; }
__device__ __forceinline__ u64 rt() { return __builtin_amdgcn_s_memrealtime(); }
__device__ __forceinline__ __amdgpu_buffer_rsrc_t mkrsrc(const void* p, unsigned bytes) {
    return __builtin_amdgcn_make_buffer_rsrc((void*)p, (short)0, (int)bytes, RSRC3);
}
#define WAIT_VM0() asm volatile("s_waitcnt vmcnt(0)" ::: "memory")
#define WAIT_VM0_B() do { asm volatile("" ::: "memory"); __builtin_amdgcn_s_waitcnt(0x0F70); asm volatile("" ::: "memory"); } while (0)
#define WAIT_LGKM0() asm volatile("s_waitcnt lgkmcnt(0)" ::: "memory")

__device__ __forceinline__ void st128(const v4u d, const __amdgpu_buffer_rsrc_t r, unsigned vo, unsigned so) {
    __builtin_amdgcn_raw_buffer_store_b128(d, r, vo, so, 17);
    asm volatile("s_nop 1" :: "v"(d) : "memory");
}

#define STAMPV(idx, val) do { } while (0)
#define STAMP(idx) do { } while (0)
#define TS(name) do { } while (0)

__device__ __forceinline__ unsigned wave_umax(unsigned v) {
    v = max(v, (unsigned)__builtin_amdgcn_update_dpp(0, (int)v, 0x111, 0xf, 0xf, true));
    v = max(v, (unsigned)__builtin_amdgcn_update_dpp(0, (int)v, 0x112, 0xf, 0xf, true));
    v = max(v, (unsigned)__builtin_amdgcn_update_dpp(0, (int)v, 0x114, 0xf, 0xf, true));
    v = max(v, (unsigned)__builtin_amdgcn_update_dpp(0, (int)v, 0x118, 0xf, 0xf, true));
    const unsigned a = __builtin_amdgcn_readlane(v, 15), b = __builtin_amdgcn_readlane(v, 31);
    const unsigned c = __builtin_amdgcn_readlane(v, 47), d = __builtin_amdgcn_readlane(v, 63);
    return max(max(a, b), max(c, d));
}

__global__ void __launch_bounds__(1024, 1) ag_gemm_kernel(Params p) {
    __shared__ __attribute__((aligned(16))) unsigned char lds[LDS_BYTES];
    LAS unsigned char* L8 = (LAS unsigned char*)lds;
    volatile LAS unsigned* CTL = (volatile LAS unsigned*)(L8 + CTL_OFF);
    const int tid = threadIdx.x;
    const int wave = __builtin_amdgcn_readfirstlane(tid >> 6);
    const int lane = tid & 63;
    const unsigned rank = (unsigned)p.rank & 7u;
    const unsigned grp = rank >> 2, mem = rank & 3u;
    const unsigned b = blockIdx.x;
    TS(tsk);
    char* const wsme = p.wsr[0];
    GAS unsigned* ctr = (GAS unsigned*)(wsme + CT_OFF + (b * 16 + wave) * 4);
    GAS unsigned* tmo = (GAS unsigned*)(wsme + TM_OFF);
    const unsigned ctl = (unsigned)CTL_OFF;

    if (b < (unsigned)(NWG - NGW)) {
        if (b - 56u < 8u) {
            if (wave != 0) return;
            const u64 t0 = rt();
            const unsigned j = b - 56u;
            const unsigned call = __builtin_amdgcn_readfirstlane(__hip_atomic_load(ctr, __ATOMIC_RELAXED, __HIP_MEMORY_SCOPE_AGENT)) + 1u;
            if (lane == 0) __hip_atomic_store(ctr, call, __ATOMIC_RELAXED, __HIP_MEMORY_SCOPE_AGENT);
            GAS u64* const dstp = (GAS u64*)(p.wsr[lane & 7] + G2_OFF + (unsigned)(lane >> 3) * 8192u + (rank * 8u + j) * 128u);
            const __amdgpu_buffer_rsrc_t rs = mkrsrc(wsme, WS_USED);
#pragma nounroll
            for (unsigned pass = opaque(WARM ? 0u : 1u); pass < 2u; pass = opaque(pass + 1u)) {
                const unsigned want = pass ? call : 0u;
                unsigned gm;
                for (unsigned it = 1;; ++it) {
                    const v4u v = __builtin_amdgcn_raw_buffer_load_b128(rs, lane * 16u, (FASTG && (it & 255u) ? G0_OFF : G1_OFF) + j * 1024u, 17);
                    gm = max(v.x, v.z);
                    if (__all(v.y >= want && v.w >= want)) break;
                    if ((it & 255u) == 0u) {
                        if (rt() - t0 > DEADLINE_TICKS ||
                            __builtin_amdgcn_readfirstlane(__hip_atomic_load(tmo, __ATOMIC_RELAXED, __HIP_MEMORY_SCOPE_AGENT)) == call) {
                            __hip_atomic_store(tmo, call, __ATOMIC_RELAXED, __HIP_MEMORY_SCOPE_AGENT);
                            gm = BAD;
                            break;
                        }
                    }
                }
                gm = wave_umax(gm);
                if (XCOPY || lane < 8) __hip_atomic_store(dstp + (pass ? 0 : 8), ((u64)call << 32) | gm, __ATOMIC_RELAXED, __HIP_MEMORY_SCOPE_SYSTEM);
            }
            STAMP(9);
            return;
        }
        const bool self = b >= 64u;
#if CU_PER_DEST == 8
        const unsigned dsel = self ? 0u : 1u + (b >> 3);
        const unsigned sub = self ? b - 64u : (b & 7u);
#else
        const unsigned dsel = self ? 0u : 1u + 2u * (b >> 3) + (b & 1u);
        const unsigned sub = self ? b - 64u : ((b & 7u) >> 1);
#endif
        if (self ? (sub >= (unsigned)SELF_CU || wave >= SELF_WV) : (dsel >= 8u || wave >= WV_PER_CU)) return;
        const unsigned ncu = self ? (unsigned)SELF_CU : (unsigned)CU_PER_DEST;
        const unsigned nwv = self ? (unsigned)(SELF_CU * SELF_WV) : (unsigned)(CU_PER_DEST * WV_PER_CU);
        const unsigned wv = (unsigned)wave * ncu + sub;
        const unsigned dst = (rank + dsel) & 7u;
        const bool same = (dst >> 2) == grp;
        const unsigned nper = self ? 16u : 5u;
        const unsigned kmul = self ? 4096u : 13108u;
        const unsigned nx = self ? 8u : same ? 5u : 3u;
        const unsigned xb0 = same ? 0u : 5u;
        const unsigned drb0 = same ? mem * 5u : 20u + mem * 3u;
        const unsigned nunits = nper * KSTEPS;
        char* const wsd = p.wsr[dsel];
        const __amdgpu_buffer_rsrc_t rd = mkrsrc(wsd, WS_USED);
        const __amdgpu_buffer_rsrc_t rx = mkrsrc(p.x, X_BYTES), rw = mkrsrc(p.w, X_BYTES);
        const unsigned xoff = (unsigned)(lane >> 3) * 8192u + (unsigned)((lane & 7) ^ (lane >> 3)) * 16u;
        const unsigned q = lane >> 4, i = lane & 15;
        const unsigned woff = q * 8192u + i * 16u;
        const unsigned wst = (i >> 3) * 4096u + (q * 16u + (i & 7u)) * 16u;
        GAS unsigned* qhead = (GAS unsigned*)(wsme + Q_OFF + dsel * 128u);
        v4u cur[8];
        auto load_unit = [&](unsigned u) {
            const unsigned ks = (u * kmul) >> 16, idx = u - ks * nper;
            if (idx < nx) {
                const unsigned so = (xb0 + idx) * (64u * 8192u) + ks * 128u;
#pragma unroll
                for (int t = 0; t < 8; ++t) cur[t] = __builtin_amdgcn_raw_buffer_load_b128(rx, xoff, so + t * 65536u, 0);
            } else {
                const unsigned wi = idx - nx;
                const unsigned so = (ks * 64u + (wi & 1u) * 32u) * 1024u + (self ? (wi >> 1) : (dst & 3u)) * 256u;
#pragma unroll
                for (int t = 0; t < 8; ++t) cur[t] = __builtin_amdgcn_raw_buffer_load_b128(rw, woff + (t & 3) * 1024u, so + (t >> 2) * 4096u, 0);
            }
        };
        load_unit(wv);
        const unsigned call_v = __hip_atomic_load(ctr, __ATOMIC_RELAXED, __HIP_MEMORY_SCOPE_AGENT) + 1u;
        if (tid < 2) CTL[tid] = 0u;
        WAIT_LGKM0();
        __builtin_amdgcn_s_barrier();
        WAIT_VM0_B();
        const unsigned call = __builtin_amdgcn_readfirstlane(call_v);
        const unsigned qbase = (call - 1u) * nunits - nwv;
        const int wbias = self ? (1 << 24) : WIN - 1;
        TS(ts0);
        unsigned u = wv;
        while (true) {
            const unsigned ks = (u * kmul) >> 16, idx = u - ks * nper;
            const bool isx = idx < nx;
            const unsigned drb = idx < 5u ? drb0 + idx : (mem & 1u) ? (idx == 5u ? 34u : 26u + idx) : 27u + idx;
            const unsigned wi = idx - nx;
            const unsigned dcb = self ? (wi >> 1) : 4u + mem, kb = wi & 1u;
            unsigned seq;
            asm volatile("s_mov_b64 exec, 1\n\t"
                         "ds_add_rtn_u32 %0, %1, %2\n\t"
                         "s_mov_b64 exec, -1"
                         : "=&v"(seq) : "v"(ctl), "v"(1u) : "memory");
            v4u o[8];
            if (isx) {
#pragma unroll
                for (int t = 0; t < 8; ++t) o[t] = cur[t];
            } else {
#pragma unroll
                for (int d = 0; d < 4; ++d) {
#pragma unroll
                    for (int e = 0; e < 4; ++e) {
                        o[2 * d][e] = __builtin_amdgcn_perm(cur[2 * e + 1][d], cur[2 * e][d], 0x05040100u);
                        o[2 * d + 1][e] = __builtin_amdgcn_perm(cur[2 * e + 1][d], cur[2 * e][d], 0x07060302u);
                    }
                }
            }
            const unsigned so = isx ? A_OFF + (ks * NRB + drb) * 8192u : B_OFF + (ks * 8u + dcb) * 16384u + kb * 8192u;
            const unsigned so2 = so + 4096u;
            const unsigned vo = (unsigned)lane * 16u, vo2 = isx ? vo : vo ^ 128u;
            const unsigned fpos = (ks & 3u) * 128u + (ks >> 2) * 8u;
            const unsigned foff = isx ? FA_OFF + (drb >> 1) * 512u + fpos + (drb & 1u) * 4u : FB_OFF + dcb * 512u + fpos + kb * 4u;
            asm volatile("s_waitcnt lgkmcnt(0)" : "+v"(seq) :: "memory");
            const int need = 8 * (int)__builtin_amdgcn_readfirstlane(seq) - wbias;
            int acked = 0;
            unsigned tk;
#pragma unroll
            for (int t = 0; t < 8; ++t) {
                if (t < WIN) {
                    while (acked < need + t) acked = (int)__builtin_amdgcn_readfirstlane(CTL[1]);
                } else {
                    asm volatile("s_waitcnt vmcnt(%2)\n\t"
                                 "s_mov_b64 exec, 1\n\t"
                                 "ds_add_u32 %0, %1 offset:4\n\t"
                                 "s_mov_b64 exec, -1" :: "v"(ctl), "v"(1u), "n"(WIN - 1 + (AP > t - WIN && AP < t ? 1 : 0)) : "memory");
                }
                if (t == AP)
                    asm volatile("s_mov_b64 exec, 1\n\t"
                                 "global_atomic_add %0, %1, %2, %3 sc0\n\t"
                                 "s_mov_b64 exec, -1"
                                 : "=&v"(tk) : "v"(0u), "v"(1u), "s"(qhead) : "memory");
                st128(o[t], rd, (t < 4 ? vo : vo2) + (t & 3) * 1024u, t < 4 ? so : so2);
            }
#define ACK(j) ".if %6 >= 8 - " #j "\n\t.if %7 > " #j "\n\ts_waitcnt vmcnt(8 - " #j ")\n\t.else\n\ts_waitcnt vmcnt(7 - " #j ")\n\t.endif\n\tds_add_u32 %5, %1 offset:4\n\t.endif\n\t"
            asm volatile("s_mov_b64 exec, 1\n\t"
                         ACK(0) ACK(1) ACK(2) ACK(3) ACK(4) ACK(5) ACK(6)
                         "s_waitcnt vmcnt(0)\n\t"
                         "global_store_dword %2, %3, %4 sc0 sc1\n\t"
                         "ds_add_u32 %5, %1 offset:4\n\t"
                         "s_mov_b64 exec, -1"
                         : "+v"(tk) : "v"(1u), "v"(foff), "v"(call), "s"(wsd), "v"(ctl), "n"(WIN), "n"(AP) : "memory");
            u = __builtin_amdgcn_readfirstlane(tk) - qbase;
            if (u >= nunits) break;
            load_unit(u);
            WAIT_VM0_B();
        }
        if (lane == 0) __hip_atomic_store(ctr, call, __ATOMIC_RELAXED, __HIP_MEMORY_SCOPE_AGENT);
        STAMP(16 + wave);
        if (wave == 0) { STAMPV(0, ts0); STAMPV(1, tsk); }
        return;
    }

    const u64 t0 = rt();
    const unsigned call_v = __hip_atomic_load(ctr, __ATOMIC_RELAXED, __HIP_MEMORY_SCOPE_AGENT) + 1u;
    if (tid < 32) CTL[tid] = tid == 18 ? BAD : 0u;
    const unsigned gb = b - (unsigned)(NWG - NGW);
    const unsigned xcd = gb & 7u, sl = gb >> 3;
    const unsigned rbp0 = (xcd >> 1) * 4u + (sl & 3u);
    const unsigned cbi = sl >> 2;
    const unsigned cb0 = (cbi & 1u) + 2u * (xcd & 1u) + 4u * (cbi >> 1);
    const bool moved = OWNROWS && cb0 >= 4u && rbp0 == ((0xfdcau >> ((cb0 - 4u) * 4u)) & 15u);
    const unsigned rbp = moved ? 16u : rbp0;
    const unsigned cb = moved ? cb0 - 4u : cb0;
    WAIT_LGKM0();
    __builtin_amdgcn_s_barrier();
    const unsigned call = __builtin_amdgcn_readfirstlane(call_v);
    if (lane == 0) __hip_atomic_store(ctr, call, __ATOMIC_RELAXED, __HIP_MEMORY_SCOPE_AGENT);

    if (wave >= 8) {
        __builtin_amdgcn_s_setprio(2);
        const unsigned s = (unsigned)wave & 3u, h = ((unsigned)wave >> 2) & 1u;
        const __amdgpu_buffer_rsrc_t rs = mkrsrc(wsme, WS_USED);
        const GAS u64* fp = (const GAS u64*)(wsme + (h ? FB_OFF + cb * 512u : FA_OFF + rbp * 512u) + s * 128u + (lane & 15) * 8u);
        const unsigned src0 = h ? B_OFF + (s * 8u + cb) * 16384u : A_OFF + (s * NRB + rbp * 2u) * 8192u;
        const unsigned sstep = h ? 4u * 8u * 16384u : 4u * NRB * 8192u;
        LAS unsigned char* dst = L8 + s * SLOT_BYTES + h * 16384u;
        const GAS u64* gp = (const GAS u64*)(wsme + G2_OFF + (XCOPY ? xcd * 8192u : 0u) + (unsigned)lane * 128u);
        unsigned known = 0;
        bool dead = false;
        if (wave == 8) STAMP(2);
#pragma nounroll
        for (unsigned pass = opaque(WARM ? 0u : 1u); pass < 2u; pass = opaque(pass + 1u)) {
            for (unsigned n = pass ? 0u : 16u; n < 16u; ++n) {
                for (unsigned it = 1; ((known >> n) & 1u) == 0u; ++it) {
                    const u64 f = __hip_atomic_load(fp, __ATOMIC_RELAXED, __HIP_MEMORY_SCOPE_SYSTEM);
                    known = (unsigned)__ballot((unsigned)f >= call && (unsigned)(f >> 32) >= call);
                    if ((it & 255u) == 0u && (rt() - t0 > DEADLINE_TICKS || CTL[8] != 0u)) { dead = true; break; }
                }
                for (unsigned it = 1; !dead && CTL[12 + s] < 8u * n; ++it) {
                    if ((it & 63u) == 0u && CTL[8] != 0u) dead = true;
                    __builtin_amdgcn_s_sleep(LSLEEP);
                }
                if (dead) break;
                asm volatile("" ::: "memory");
                const unsigned sa = src0 + n * sstep;
#pragma unroll
                for (int g = 0; g < 4; ++g) {
                    __builtin_amdgcn_raw_ptr_buffer_load_lds(rs, dst + g * 4096, 16, lane * 16u, sa + g * 4096u, 0, 17);
                    __builtin_amdgcn_raw_ptr_buffer_load_lds(rs, dst + g * 4096, 16, lane * 16u, sa + g * 4096u, 1024, 17);
                    __builtin_amdgcn_raw_ptr_buffer_load_lds(rs, dst + g * 4096, 16, lane * 16u, sa + g * 4096u, 2048, 17);
                    __builtin_amdgcn_raw_ptr_buffer_load_lds(rs, dst + g * 4096, 16, lane * 16u, sa + g * 4096u, 3072, 17);
                }
                WAIT_VM0();
                CTL[s * 2u + h] = n + 1u;
            }
            if (dead) {
                __hip_atomic_store(tmo, call, __ATOMIC_RELAXED, __HIP_MEMORY_SCOPE_AGENT);
                CTL[8] = 1u; CTL[9] = 1u;
            }
            if (wave == 8) {
                const unsigned want = pass ? call : 0u;
                unsigned best = 0u, sbits = 0x80000000u;
                u64 v = __hip_atomic_load(gp, __ATOMIC_RELAXED, __HIP_MEMORY_SCOPE_SYSTEM);
                for (unsigned it = 1;; ++it) {
                    const bool ok = (unsigned)(v >> 32) >= want;
                    const unsigned gm = ok ? (unsigned)v | (pass ^ 1u) : 0u;
                    const bool all = __all(ok), newer = __any(gm > best);
                    if (all && !newer) break;
                    v = __hip_atomic_load(gp, __ATOMIC_RELAXED, __HIP_MEMORY_SCOPE_SYSTEM);
                    if (newer) {
                        best = wave_umax(gm);
                        sbits = best == BAD ? BAD : __float_as_uint(127.0f / __uint_as_float(best)) | (best >= 0x7f800000u ? 0x80000000u : 0u);
                    }
                    if (all) break;
                    if ((it & 63u) == 0u) {
                        if (rt() - t0 > DEADLINE_TICKS ||
                            __builtin_amdgcn_readfirstlane(__hip_atomic_load(tmo, __ATOMIC_RELAXED, __HIP_MEMORY_SCOPE_AGENT)) == call) {
                            __hip_atomic_store(tmo, call, __ATOMIC_RELAXED, __HIP_MEMORY_SCOPE_AGENT);
                            sbits = BAD;
                            break;
                        }
                    }
                }
                CTL[pass ? 16u : 19u] = sbits;
            }
            WAIT_LGKM0();
        }
        return;
    }

    const int wm = wave >> 1, wn = wave & 1;
    const int g = lane >> 4, i = lane & 15;
    const unsigned rb = rbp0 * 2u + ((unsigned)wm >> 1);
    const unsigned sidx = rb < 20u ? rb / 5u : 4u + (rb - 20u) / 3u;
    const unsigned sblk = rb < 20u ? rb % 5u : 5u + (rb - 20u) % 3u;
    const unsigned rloc = ((unsigned)wm & 1u) * 32u;
    const unsigned grow = moved ? rank * 512u + (5u + (mem & 1u) + ((unsigned)wm >> 1)) * 64u + rloc
                                : ((sidx + grp * 4u) & 7u) * 512u + sblk * 64u + rloc;
    const unsigned prow = ((sidx + grp * 4u + 4u) & 7u) * 512u + sblk * 64u + rloc;
    const bool remote = cb >= 4u;
    const bool watch = cb0 < 4u && !(OWNROWS && rbp0 == ((0xfdcau >> (mem * 4u)) & 15u));
    const unsigned ocol = (remote ? mem : cb) * 128u + (unsigned)wn * 64u;
    char* const outp = p.outr[remote ? (cb - mem) & 7u : 0u];
    const unsigned pvo = (unsigned)(lane >> 2) * 512u + (unsigned)(lane & 3) * 16u;
    GAS u64* const g1 = (GAS u64*)(wsme + G1_OFF + xcd * 1024u + (sl * 8u + (unsigned)wave) * 8u);
    f32x4 acc[2][4];
#pragma unroll
    for (int m = 0; m < 2; ++m)
#pragma unroll
        for (int f = 0; f < 4; ++f) acc[m][f] = (f32x4){0.f, 0.f, 0.f, 0.f};
    const unsigned a_off0 = (unsigned)(32 * wm + i) * 128u + (unsigned)((g ^ (i & 7)) * 16);
    const unsigned a_off1 = (unsigned)(32 * wm + i) * 128u + (unsigned)(((4 + g) ^ (i & 7)) * 16);
    const unsigned b_off = 16384u + (unsigned)(i & 1) * 4096u + (unsigned)(g * 16 + ((8 * wn + (i >> 1)) ^ (8 * (i & 1)))) * 16u;
    bool dead = false;
    unsigned mycnt = 0;
    unsigned last = 3;
    const volatile LAS v2u* fullp = (const volatile LAS v2u*)(L8 + CTL_OFF + (lane < 4 ? lane : 4) * 8);
#pragma nounroll
    for (unsigned pass = opaque(0u); pass < 2u; pass = opaque(pass + 1u)) {
    const __amdgpu_buffer_rsrc_t ro = mkrsrc(outp, pass || watch ? OUT_BYTES : 0u);
    dead = false;
    for (unsigned n = pass ? 0u : (unsigned)KSTEPS; n < (unsigned)KSTEPS; ++n) {
        unsigned m;
        while (true) {
            const v2u f = *fullp;
            m = (unsigned)__ballot(min(f.x, f.y) > mycnt) & 31u;
            if (m) break;
            __builtin_amdgcn_s_sleep(CSLEEP);
        }
        if (m & 16u) { dead = true; break; }
        const unsigned s = (last + 1u + (unsigned)__builtin_ctz((m | (m << 4)) >> (last + 1u))) & 3u;
        last = s;
        asm volatile("" ::: "memory");
        const LAS unsigned char* sp = L8 + s * SLOT_BYTES;
        v4u af[2][2], bf[2][4];
#pragma unroll
        for (int kb = 0; kb < 2; ++kb) {
            af[kb][0] = *(const LAS v4u*)(sp + (kb ? a_off1 : a_off0));
#pragma unroll
            for (int f = 0; f < 4; ++f) bf[kb][f] = *(const LAS v4u*)(sp + b_off + kb * 8192 + f * 1024);
            af[kb][1] = *(const LAS v4u*)(sp + (kb ? a_off1 : a_off0) + 2048);
        }
        mycnt += ((unsigned)lane == s) ? 1u : 0u;
        const unsigned fa = ctl + 48u + s * 4u;
        __builtin_amdgcn_sched_barrier(0);
#pragma unroll
        for (int mm = 0; mm < 2; ++mm)
#pragma unroll
            for (int f = 0; f < 4; ++f)
                acc[mm][f] = __builtin_amdgcn_mfma_f32_16x16x32_bf16(__builtin_bit_cast(bf16x8, af[0][mm]),
                                                                     __builtin_bit_cast(bf16x8, bf[0][f]), acc[mm][f], 0, 0, 0);
        __builtin_amdgcn_sched_barrier(0);
        asm volatile("s_waitcnt lgkmcnt(0)\n\t"
                     "s_mov_b64 exec, 1\n\t"
                     "ds_add_u32 %6, %7\n\t"
                     "s_mov_b64 exec, -1"
                     : "+v"(af[1][0]), "+v"(af[1][1]), "+v"(bf[1][0]), "+v"(bf[1][1]), "+v"(bf[1][2]), "+v"(bf[1][3])
                     : "v"(fa), "v"(1u) : "memory");
        __builtin_amdgcn_sched_barrier(0);
#pragma unroll
        for (int kb = COVL; kb < 2; ++kb)
#pragma unroll
            for (int mm = 0; mm < 2; ++mm)
#pragma unroll
                for (int f = 0; f < 4; ++f)
                    acc[mm][f] = __builtin_amdgcn_mfma_f32_16x16x32_bf16(__builtin_bit_cast(bf16x8, af[kb][mm]),
                                                                         __builtin_bit_cast(bf16x8, bf[kb][f]), acc[mm][f], 0, 0, 0);
    }
    TS(ts5);

    {
        float mx = 0.f;
#pragma unroll
        for (int m = 0; m < 2; ++m)
#pragma unroll
            for (int f = 0; f < 4; ++f)
#pragma unroll
                for (int r = 0; r < 4; ++r) mx = fmaxf(mx, acc[m][f][r]);
        const unsigned wmx = dead ? BAD : wave_umax(__float_as_uint(mx));
        if (lane == 0) {
            if (FASTG) __hip_atomic_store(g1 + (G0_OFF - G1_OFF) / 8, ((u64)(call - 1u + pass) << 32) | wmx, __ATOMIC_RELAXED, __HIP_MEMORY_SCOPE_WORKGROUP);
            __hip_atomic_store(g1, ((u64)(call - 1u + pass) << 32) | wmx, __ATOMIC_RELAXED, __HIP_MEMORY_SCOPE_SYSTEM);
        }
    }
    TS(ts6);
    asm volatile("s_waitcnt vmcnt(0)" : "+v"(acc[0][0]), "+v"(acc[0][1]), "+v"(acc[0][2]), "+v"(acc[0][3]),
                      "+v"(acc[1][0]), "+v"(acc[1][1]), "+v"(acc[1][2]), "+v"(acc[1][3]) :: "memory");
#pragma unroll
    for (int m = 0; m < 2; ++m)
#pragma unroll
        for (int f = 0; f < 4; ++f)
#pragma unroll
            for (int r = 0; r < 4; ++r) acc[m][f][r] = __int_as_float(max(__float_as_int(acc[m][f][r]), 0));
    asm volatile("" : "+v"(acc[0][0]), "+v"(acc[0][1]), "+v"(acc[0][2]), "+v"(acc[0][3]),
                      "+v"(acc[1][0]), "+v"(acc[1][1]), "+v"(acc[1][2]), "+v"(acc[1][3]) :: "memory");
    unsigned sb;
    do { sb = __builtin_amdgcn_readfirstlane(CTL[pass ? 16u : 18u]); } while (sb == 0u);
    TS(ts7);
    dead = sb == BAD;
    const float inv = dead ? 0.f : __uint_as_float(sb);
    const float magic = dead ? 12583040.f : 12582912.f;

    {
        const unsigned ovo = (unsigned)g * 2048u + (unsigned)i * 4u;
        const int need = remote && pass ? 2 * wave + 1 - OUT_WIN : -2;
        const unsigned orow = pass ? grow : prow;
        int acked = 0;
#pragma unroll
        for (int m = 0; m < 2; ++m) {
            unsigned pk[4];
            const f32x2 iv = {inv, inv}, mg = {magic, magic};
            f32x2 z[4][2];
#pragma unroll
            for (int f = 0; f < 4; ++f) {
                z[f][0] = __builtin_elementwise_fma((f32x2){acc[m][f][0], acc[m][f][1]}, iv, mg);
                z[f][1] = __builtin_elementwise_fma((f32x2){acc[m][f][2], acc[m][f][3]}, iv, mg);
            }
#pragma unroll
            for (int r = 0; r < 4; ++r) {
                const unsigned lo = __builtin_amdgcn_perm(__float_as_uint(z[1][r >> 1][r & 1]), __float_as_uint(z[0][r >> 1][r & 1]), 0x0c0c0400u);
                const unsigned hi = __builtin_amdgcn_perm(__float_as_uint(z[3][r >> 1][r & 1]), __float_as_uint(z[2][r >> 1][r & 1]), 0x04000c0cu);
                pk[r] = lo | hi;
            }
            asm volatile("" : "+v"(pk[0]), "+v"(pk[1]), "+v"(pk[2]), "+v"(pk[3]) :: "memory");
            while (acked < need + m) acked = (int)__builtin_amdgcn_readfirstlane(CTL[17]);
#pragma unroll
            for (int r = 0; r < 4; ++r)
                __builtin_amdgcn_raw_buffer_store_b32(pk[r], ro, ovo + r * 512u, (orow + 16u * m) * 512u + ocol, 17);
        }
        if (remote)
            asm volatile("s_mov_b64 exec, 1\n\t"
                         "s_waitcnt vmcnt(4)\n\tds_add_u32 %0, %1 offset:68\n\t"
                         "s_waitcnt vmcnt(0)\n\tds_add_u32 %0, %1 offset:68\n\t"
                         "s_mov_b64 exec, -1" :: "v"(ctl), "v"(pass) : "memory");
    }
    if (watch) {
        for (unsigned it = 1;; ++it) {
            const v4u a0 = __builtin_amdgcn_raw_buffer_load_b128(ro, pvo, prow * 512u + ocol, 17);
            const v4u a1 = __builtin_amdgcn_raw_buffer_load_b128(ro, pvo, (prow + 16u) * 512u + ocol, 17);
            const v4u a = a0 | a1;
            if (__all(((a.x | a.y | a.z | a.w) & 0x80808080u) == 0u) || dead) break;
            if ((it & 63u) == 0u) {
                if (rt() - t0 > DEADLINE_TICKS ||
                    __builtin_amdgcn_readfirstlane(__hip_atomic_load(tmo, __ATOMIC_RELAXED, __HIP_MEMORY_SCOPE_AGENT)) == call) {
                    __hip_atomic_store(tmo, call, __ATOMIC_RELAXED, __HIP_MEMORY_SCOPE_AGENT);
                    if (lane == 0) __builtin_amdgcn_raw_buffer_store_b8((unsigned char)0x80, ro, 0u, grow * 512u + ocol, 17);
                    break;
                }
            }
        }
    }
    if (pass == 0u) { WAIT_VM0(); continue; }
    }
}

extern "C" void kernel_launch(int rank, int nranks, ncclComm_t comm, hipStream_t stream,
                              void* const* d_in, const int* in_sizes, int n_in,
                              void* d_out, int out_size, const int* in_dtypes, int out_dtype,
                              void* const* d_out_all, void* const* d_ws_all, size_t ws_bytes) {
    if (nranks != NR || n_in < 2 || ws_bytes < (size_t)WS_USED + 4096u || (unsigned)rank >= (unsigned)NR ||
        in_sizes[0] != 512 * 4096 || in_sizes[1] != 4096 * 512 || out_size != 4096 * 512) {
        fprintf(stderr, "unexpected problem configuration\n");
        return;
    }
    Params p;
    p.x = d_in[0];
    p.w = d_in[1];
    for (int d = 0; d < NR; ++d) { p.wsr[d] = (char*)d_ws_all[(rank + d) & 7]; p.outr[d] = (char*)d_out_all[(rank + d) & 7]; }
    p.rank = rank;
    ag_gemm_kernel<<<dim3(NWG), dim3(1024), 0, stream>>>(p);
}
```

```cpp
#include <hip/hip_runtime.h>
#include <rccl/rccl.h>
#include <cstdio>
#include <cstdint>

#define LAS __attribute__((address_space(3)))
#define GAS __attribute__((address_space(1)))
typedef unsigned v4u __attribute__((ext_vector_type(4)));
typedef unsigned v2u __attribute__((ext_vector_type(2)));
typedef float f32x4 __attribute__((ext_vector_type(4)));
typedef float f32x2 __attribute__((ext_vector_type(2)));
typedef __bf16 bf16x8 __attribute__((ext_vector_type(8)));
typedef unsigned long long u64;

#ifndef CU_PER_DEST
#define CU_PER_DEST 8
#endif
#define WV_PER_CU 3
#define WIN 6
#define SELF_CU 16
#define SELF_WV 4
#define OUT_WIN 4
#define XCOPY 1
#define WLIN 1
#define FASTG 1
#define COVL 1
#define WARM 1
#define AP 4
#define CSLEEP 1
#define LSLEEP 1
#define GWARM 1
#define CPRIO 0
#define AGW 1
#define GAW 1
#define FPD 1
#define FPS 4
#define FPN 15
#define DPD 1
#define DPS 4
#define MPOS 0
#define FDELAY 0
#define FSLEEP 0
#define GSLEEP 16
#define GTHR 48
#define ASLEEP 0
#define TWARM 0
#define MALL 0
#define AAUX 17
#define OAUX 17
#define OWNROWS 1

constexpr int NR = 8;
constexpr int NWG = 256;
constexpr int NGW = 128;
constexpr int KSTEPS = 64;
constexpr int NSLOT = 4;
constexpr unsigned NRB = 36;
constexpr unsigned A_OFF = 0;
constexpr unsigned B_OFF = 18u << 20;
constexpr unsigned F_OFF = 26u << 20;
constexpr unsigned FA_OFF = F_OFF;
constexpr unsigned FB_OFF = F_OFF + 12288;
constexpr unsigned Q_OFF = F_OFF + 16384;
constexpr unsigned TM_OFF = F_OFF + 18432;
constexpr unsigned G1_OFF = F_OFF + 24576;
constexpr unsigned CT_OFF = F_OFF + 32768;
constexpr unsigned G0_OFF = F_OFF + 49152;
constexpr unsigned G2_OFF = F_OFF + 65536;
constexpr unsigned DBG_OFF = F_OFF + 131072;
constexpr unsigned M_OFF = F_OFF + (512u << 10);
constexpr unsigned D_OFF = 27u << 20;
constexpr unsigned WS_USED = 28u << 20;
constexpr unsigned X_BYTES = 512u * 4096u * 2u;
constexpr unsigned OUT_BYTES = 4096u * 512u;
constexpr int SLOT_BYTES = 32768;
constexpr int CTL_OFF = NSLOT * SLOT_BYTES;
constexpr int LDS_BYTES = CTL_OFF + 128;
constexpr u64 DEADLINE_TICKS = 40000000ull;
constexpr int RSRC3 = 0x00020000;
constexpr unsigned BAD = 0xffffffffu;
static_assert(LDS_BYTES <= 160 * 1024, "LDS");

struct Params {
    const void* x;
    const void* w;
    char* wsr[NR];
    void* out;
    int rank;
};

__device__ __forceinline__ unsigned opaque(unsigned v) { asm volatile("" : "+s"(v)); return v; }
__device__ __forceinline__ u64 rt() { return __builtin_amdgcn_s_memrealtime(); }
__device__ __forceinline__ __amdgpu_buffer_rsrc_t mkrsrc(const void* p, unsigned bytes) {
    return __builtin_amdgcn_make_buffer_rsrc((void*)p, (short)0, (int)bytes, RSRC3);
}
#define WAIT_VM0() asm volatile("s_waitcnt vmcnt(0)" ::: "memory")
#define WAIT_VM0_B() do { asm volatile("" ::: "memory"); __builtin_amdgcn_s_waitcnt(0x0F70); asm volatile("" ::: "memory"); } while (0)
#define WAIT_LGKM0() asm volatile("s_waitcnt lgkmcnt(0)" ::: "memory")

__device__ __forceinline__ void st128(const v4u d, const __amdgpu_buffer_rsrc_t r, unsigned vo, unsigned so) {
    __builtin_amdgcn_raw_buffer_store_b128(d, r, vo, so, 17);
    asm volatile("s_nop 1" :: "v"(d) : "memory");
}

#define STAMPV(idx, val) do { } while (0)
#define STAMP(idx) do { } while (0)
#define TS(name) do { } while (0)

__device__ __forceinline__ unsigned wave_umax(unsigned v) {
    v = max(v, (unsigned)__builtin_amdgcn_update_dpp(0, (int)v, 0x111, 0xf, 0xf, true));
    v = max(v, (unsigned)__builtin_amdgcn_update_dpp(0, (int)v, 0x112, 0xf, 0xf, true));
    v = max(v, (unsigned)__builtin_amdgcn_update_dpp(0, (int)v, 0x114, 0xf, 0xf, true));
    v = max(v, (unsigned)__builtin_amdgcn_update_dpp(0, (int)v, 0x118, 0xf, 0xf, true));
    const unsigned a = __builtin_amdgcn_readlane(v, 15), b = __builtin_amdgcn_readlane(v, 31);
    const unsigned c = __builtin_amdgcn_readlane(v, 47), d = __builtin_amdgcn_readlane(v, 63);
    return max(max(a, b), max(c, d));
}

__device__ __forceinline__ unsigned wave_scan(unsigned v) {
    v += (unsigned)__builtin_amdgcn_update_dpp(0, (int)v, 0x111, 0xf, 0xf, true);
    v += (unsigned)__builtin_amdgcn_update_dpp(0, (int)v, 0x112, 0xf, 0xf, true);
    v += (unsigned)__builtin_amdgcn_update_dpp(0, (int)v, 0x114, 0xf, 0xf, true);
    v += (unsigned)__builtin_amdgcn_update_dpp(0, (int)v, 0x118, 0xf, 0xf, true);
    v += (unsigned)__builtin_amdgcn_update_dpp(0, (int)v, 0x142, 0xa, 0xf, false);
    v += (unsigned)__builtin_amdgcn_update_dpp(0, (int)v, 0x143, 0xc, 0xf, false);
    return v;
}

__device__ __forceinline__ unsigned positions(unsigned mk, unsigned (&o)[8]) {
    unsigned x = mk;
    x -= (x >> 1) & 0x55555555u;
    x = (x & 0x33333333u) + ((x >> 2) & 0x33333333u);
    unsigned base = 0;
    unsigned c[2], e[2], cl[2], el[2];
    c[0] = x & 0x0f0f0f0fu;
    c[1] = (x >> 4) & 0x0f0f0f0fu;
#pragma unroll
    for (int h = 0; h < 2; ++h) {
        e[h] = wave_scan(c[h]) - c[h];
        cl[h] = __builtin_amdgcn_readlane(c[h], 63);
        el[h] = __builtin_amdgcn_readlane(e[h], 63);
    }
#pragma unroll
    for (int j = 0; j < 8; ++j) {
        o[j] = base + ((e[j & 1] >> (8 * (j >> 1))) & 255u);
        base += ((cl[j & 1] >> (8 * (j >> 1))) & 255u) + ((el[j & 1] >> (8 * (j >> 1))) & 255u);
    }
    return base;
}

__device__ __forceinline__ unsigned quant4(const f32x4 d, float inv, float magic) {
    const f32x2 iv = {inv, inv}, mg = {magic, magic};
    const f32x2 a = __builtin_elementwise_fma((f32x2){d[0], d[1]}, iv, mg), b = __builtin_elementwise_fma((f32x2){d[2], d[3]}, iv, mg);
    return __builtin_amdgcn_perm(__float_as_uint(a[1]), __float_as_uint(a[0]), 0x0c0c0400u) |
           __builtin_amdgcn_perm(__float_as_uint(b[1]), __float_as_uint(b[0]), 0x04000c0cu);
}

__global__ void __launch_bounds__(1024, 1) ag_gemm_kernel(Params p) {
    __shared__ __attribute__((aligned(16))) unsigned char lds[LDS_BYTES];
    LAS unsigned char* L8 = (LAS unsigned char*)lds;
    volatile LAS unsigned* CTL = (volatile LAS unsigned*)(L8 + CTL_OFF);
    const int tid = threadIdx.x;
    const int wave = __builtin_amdgcn_readfirstlane(tid >> 6);
    const int lane = tid & 63;
    const unsigned rank = (unsigned)p.rank & 7u;
    const unsigned grp = rank >> 2, mem = rank & 3u;
    const unsigned b = blockIdx.x;
    TS(tsk);
    char* const wsme = p.wsr[0];
    GAS unsigned* ctr = (GAS unsigned*)(wsme + CT_OFF + (b * 16 + wave) * 4);
    GAS unsigned* tmo = (GAS unsigned*)(wsme + TM_OFF);
    const unsigned ctl = (unsigned)CTL_OFF;

    if (b < (unsigned)(NWG - NGW)) {
        if (b - 56u < 8u) {
            if (wave >= AGW) return;
            const u64 t0 = rt();
            const unsigned j = b - 56u;
            const unsigned call = __builtin_amdgcn_readfirstlane(__hip_atomic_load(ctr, __ATOMIC_RELAXED, __HIP_MEMORY_SCOPE_AGENT)) + 1u;
            if (lane == 0) __hip_atomic_store(ctr, call, __ATOMIC_RELAXED, __HIP_MEMORY_SCOPE_AGENT);
            GAS u64* const dstp = (GAS u64*)(p.wsr[lane & 7] + G2_OFF + (unsigned)(lane >> 3) * 8192u + (rank * 8u + j) * 128u);
            const __amdgpu_buffer_rsrc_t rs = mkrsrc(wsme, WS_USED);
#pragma nounroll
            for (unsigned pass = opaque(WARM ? 0u : 1u); pass < 2u; pass = opaque(pass + 1u)) {
                const unsigned want = pass ? call : 0u;
                unsigned gm;
                for (unsigned it = 1;; ++it) {
                    const v4u v = __builtin_amdgcn_raw_buffer_load_b128(rs, lane * 16u, (FASTG && (it & 255u) ? G0_OFF : G1_OFF) + j * 1024u, AAUX);
                    gm = max(v.x, v.z);
                    if (__all(v.y >= want && v.w >= want)) break;
                    if (ASLEEP) __builtin_amdgcn_s_sleep(ASLEEP);
                    if ((it & 255u) == 0u) {
                        if (rt() - t0 > DEADLINE_TICKS ||
                            __builtin_amdgcn_readfirstlane(__hip_atomic_load(tmo, __ATOMIC_RELAXED, __HIP_MEMORY_SCOPE_AGENT)) == call) {
                            __hip_atomic_store(tmo, call, __ATOMIC_RELAXED, __HIP_MEMORY_SCOPE_AGENT);
                            gm = BAD;
                            break;
                        }
                    }
                }
                gm = wave_umax(gm);
                if (XCOPY || lane < 8) __hip_atomic_store(dstp + (pass ? 0 : 8), ((u64)call << 32) | gm, __ATOMIC_RELAXED, __HIP_MEMORY_SCOPE_SYSTEM);
            }
            STAMP(9 + wave);
            return;
        }
        const bool self = b >= 64u;
#if CU_PER_DEST == 8
        const unsigned dsel = self ? 0u : 1u + (b >> 3);
        const unsigned sub = self ? b - 64u : (b & 7u);
#else
        const unsigned dsel = self ? 0u : 1u + 2u * (b >> 3) + (b & 1u);
        const unsigned sub = self ? b - 64u : ((b & 7u) >> 1);
#endif
        if (self ? (sub >= (unsigned)SELF_CU || wave >= SELF_WV) : (dsel >= 8u || wave >= WV_PER_CU)) return;
        const unsigned ncu = self ? (unsigned)SELF_CU : (unsigned)CU_PER_DEST;
        const unsigned nwv = self ? (unsigned)(SELF_CU * SELF_WV) : (unsigned)(CU_PER_DEST * WV_PER_CU);
        const unsigned wv = (unsigned)wave * ncu + sub;
        const unsigned dst = (rank + dsel) & 7u;
        const bool same = (dst >> 2) == grp;
        const unsigned nper = self ? 16u : 5u;
        const unsigned kmul = self ? 4096u : 13108u;
        const unsigned nx = self ? 8u : same ? 5u : 3u;
        const unsigned xb0 = same ? 0u : 5u;
        const unsigned drb0 = same ? mem * 5u : 20u + mem * 3u;
        const unsigned nunits = nper * KSTEPS;
        char* const wsd = p.wsr[dsel];
        const __amdgpu_buffer_rsrc_t rd = mkrsrc(wsd, WS_USED);
        const __amdgpu_buffer_rsrc_t rx = mkrsrc(p.x, X_BYTES), rw = mkrsrc(p.w, X_BYTES);
        const unsigned xoff = (unsigned)(lane >> 3) * 8192u + (unsigned)((lane & 7) ^ (lane >> 3)) * 16u;
        const unsigned q = lane >> 4, i = lane & 15;
        const unsigned woff = q * 8192u + i * 16u;
        const unsigned wst = (i >> 3) * 4096u + (q * 16u + (i & 7u)) * 16u;
        GAS unsigned* qhead = (GAS unsigned*)(wsme + Q_OFF + dsel * 128u);
        v4u cur[8];
        auto load_unit = [&](unsigned u) {
            const unsigned ks = (u * kmul) >> 16, idx = u - ks * nper;
            if (idx < nx) {
                const unsigned so = (xb0 + idx) * (64u * 8192u) + ks * 128u;
#pragma unroll
                for (int t = 0; t < 8; ++t) cur[t] = __builtin_amdgcn_raw_buffer_load_b128(rx, xoff, so + t * 65536u, 0);
            } else {
                const unsigned wi = idx - nx;
                const unsigned so = (ks * 64u + (wi & 1u) * 32u) * 1024u + (self ? (wi >> 1) : (dst & 3u)) * 256u;
#pragma unroll
                for (int t = 0; t < 8; ++t) cur[t] = __builtin_amdgcn_raw_buffer_load_b128(rw, woff + (t & 3) * 1024u, so + (t >> 2) * 4096u, 0);
            }
        };
        load_unit(wv);
        const unsigned call_v = __hip_atomic_load(ctr, __ATOMIC_RELAXED, __HIP_MEMORY_SCOPE_AGENT) + 1u;
        if (tid < 2) CTL[tid] = 0u;
        WAIT_LGKM0();
        __builtin_amdgcn_s_barrier();
        WAIT_VM0_B();
        const unsigned call = __builtin_amdgcn_readfirstlane(call_v);
        const unsigned qbase = (call - 1u) * nunits - nwv;
        const int wbias = self ? (1 << 24) : WIN - 1;
        TS(ts0);
        unsigned u = wv;
        while (true) {
            const unsigned ks = (u * kmul) >> 16, idx = u - ks * nper;
            const bool isx = idx < nx;
            const unsigned drb = idx < 5u ? drb0 + idx : (mem & 1u) ? (idx == 5u ? 34u : 26u + idx) : 27u + idx;
            const unsigned wi = idx - nx;
            const unsigned dcb = self ? (wi >> 1) : 4u + mem, kb = wi & 1u;
            unsigned seq;
            asm volatile("s_mov_b64 exec, 1\n\t"
                         "ds_add_rtn_u32 %0, %1, %2\n\t"
                         "s_mov_b64 exec, -1"
                         : "=&v"(seq) : "v"(ctl), "v"(1u) : "memory");
            v4u o[8];
            if (isx) {
#pragma unroll
                for (int t = 0; t < 8; ++t) o[t] = cur[t];
            } else {
#pragma unroll
                for (int d = 0; d < 4; ++d) {
#pragma unroll
                    for (int e = 0; e < 4; ++e) {
                        o[2 * d][e] = __builtin_amdgcn_perm(cur[2 * e + 1][d], cur[2 * e][d], 0x05040100u);
                        o[2 * d + 1][e] = __builtin_amdgcn_perm(cur[2 * e + 1][d], cur[2 * e][d], 0x07060302u);
                    }
                }
            }
            const unsigned so = isx ? A_OFF + (ks * NRB + drb) * 8192u : B_OFF + (ks * 8u + dcb) * 16384u + kb * 8192u;
            const unsigned so2 = so + 4096u;
            const unsigned vo = (unsigned)lane * 16u, vo2 = isx ? vo : vo ^ 128u;
            const unsigned fpos = (ks & 3u) * 128u + (ks >> 2) * 8u;
            const unsigned foff = isx ? FA_OFF + (drb >> 1) * 512u + fpos + (drb & 1u) * 4u : FB_OFF + dcb * 512u + fpos + kb * 4u;
            asm volatile("s_waitcnt lgkmcnt(0)" : "+v"(seq) :: "memory");
            const int need = 8 * (int)__builtin_amdgcn_readfirstlane(seq) - wbias;
            int acked = 0;
            unsigned tk;
#pragma unroll
            for (int t = 0; t < 8; ++t) {
                if (t < WIN) {
                    while (acked < need + t) acked = (int)__builtin_amdgcn_readfirstlane(CTL[1]);
                } else {
                    asm volatile("s_waitcnt vmcnt(%2)\n\t"
                                 "s_mov_b64 exec, 1\n\t"
                                 "ds_add_u32 %0, %1 offset:4\n\t"
                                 "s_mov_b64 exec, -1" :: "v"(ctl), "v"(1u), "n"(WIN - 1 + (AP > t - WIN && AP < t ? 1 : 0)) : "memory");
                }
                if (t == AP)
                    asm volatile("s_mov_b64 exec, 1\n\t"
                                 "global_atomic_add %0, %1, %2, %3 sc0\n\t"
                                 "s_mov_b64 exec, -1"
                                 : "=&v"(tk) : "v"(0u), "v"(1u), "s"(qhead) : "memory");
                st128(o[t], rd, (t < 4 ? vo : vo2) + (t & 3) * 1024u, t < 4 ? so : so2);
            }
#define ACK(j) ".if %6 >= 8 - " #j "\n\t.if %7 > " #j "\n\ts_waitcnt vmcnt(8 - " #j ")\n\t.else\n\ts_waitcnt vmcnt(7 - " #j ")\n\t.endif\n\tds_add_u32 %5, %1 offset:4\n\t.endif\n\t"
            asm volatile("s_mov_b64 exec, 1\n\t"
                         ACK(0) ACK(1) ACK(2) ACK(3) ACK(4) ACK(5) ACK(6)
                         "s_waitcnt vmcnt(0)\n\t"
                         "global_store_dword %2, %3, %4 sc0 sc1\n\t"
                         "ds_add_u32 %5, %1 offset:4\n\t"
                         "s_mov_b64 exec, -1"
                         : "+v"(tk) : "v"(1u), "v"(foff), "v"(call), "s"(wsd), "v"(ctl), "n"(WIN), "n"(AP) : "memory");
            u = __builtin_amdgcn_readfirstlane(tk) - qbase;
            if (u >= nunits) break;
            load_unit(u);
            WAIT_VM0_B();
        }
        if (lane == 0) __hip_atomic_store(ctr, call, __ATOMIC_RELAXED, __HIP_MEMORY_SCOPE_AGENT);
        STAMP(16 + wave);
        if (wave == 0) { STAMPV(0, ts0); STAMPV(1, tsk); }
        return;
    }

    const u64 t0 = rt();
    const unsigned call_v = __hip_atomic_load(ctr, __ATOMIC_RELAXED, __HIP_MEMORY_SCOPE_AGENT) + 1u;
    if (tid < 32) CTL[tid] = tid == 18 ? BAD : 0u;
    const unsigned gb = b - (unsigned)(NWG - NGW);
    const unsigned xcd = gb & 7u, sl = gb >> 3;
    const unsigned rbp0 = (xcd >> 1) * 4u + (sl & 3u);
    const unsigned cbi = sl >> 2;
    const unsigned cb0 = (cbi & 1u) + 2u * (xcd & 1u) + 4u * (cbi >> 1);
    const bool moved = OWNROWS && cb0 >= 4u && rbp0 == ((0xfdcau >> ((cb0 - 4u) * 4u)) & 15u);
    const unsigned rbp = moved ? 16u : rbp0;
    const unsigned cb = moved ? cb0 - 4u : cb0;
    WAIT_LGKM0();
    __builtin_amdgcn_s_barrier();
    const unsigned call = __builtin_amdgcn_readfirstlane(call_v);
    if (lane == 0) __hip_atomic_store(ctr, call, __ATOMIC_RELAXED, __HIP_MEMORY_SCOPE_AGENT);

    if (wave >= 8) {
        __builtin_amdgcn_s_setprio(2);
        const unsigned s = (unsigned)wave & 3u, h = ((unsigned)wave >> 2) & 1u;
        const __amdgpu_buffer_rsrc_t rs = mkrsrc(wsme, WS_USED);
        const GAS u64* fp = (const GAS u64*)(wsme + (h ? FB_OFF + cb * 512u : FA_OFF + rbp * 512u) + s * 128u + (lane & 15) * 8u);
        const unsigned src0 = h ? B_OFF + (s * 8u + cb) * 16384u : A_OFF + (s * NRB + rbp * 2u) * 8192u;
        const unsigned sstep = h ? 4u * 8u * 16384u : 4u * NRB * 8192u;
        LAS unsigned char* dst = L8 + s * SLOT_BYTES + h * 16384u;
        const GAS u64* gp = (const GAS u64*)(wsme + G2_OFF + (XCOPY ? xcd * 8192u : 0u) + (unsigned)lane * 128u);
        unsigned known = 0;
        bool dead = false;
        if (wave == 8) STAMP(2);
#pragma nounroll
        for (unsigned pass = opaque(WARM ? 0u : 1u); pass < 2u; pass = opaque(pass + 1u)) {
            for (unsigned n = pass ? 0u : 16u; n < 16u; ++n) {
                if (FDELAY && n > 0u && ((known >> n) & 1u) == 0u && !(h && cb < 4u) && !moved) __builtin_amdgcn_s_sleep(FDELAY);
                for (unsigned it = 1; !dead && ((known >> n) & 1u) == 0u; ++it) {
                    if (FSLEEP && it > 1u) __builtin_amdgcn_s_sleep(FSLEEP);
                    const u64 f = __hip_atomic_load(fp, __ATOMIC_RELAXED, __HIP_MEMORY_SCOPE_SYSTEM);
                    known = (unsigned)__ballot((unsigned)f >= call && (unsigned)(f >> 32) >= call);
                    if ((it & 255u) == 0u && (rt() - t0 > DEADLINE_TICKS || CTL[8] != 0u)) { dead = true; break; }
                }
                for (unsigned it = 1; !dead && CTL[12 + s] < 8u * n; ++it) {
                    if ((it & 63u) == 0u && CTL[8] != 0u) dead = true;
                    __builtin_amdgcn_s_sleep(LSLEEP);
                }
                if (dead) break;
                asm volatile("" ::: "memory");
                const unsigned sa = src0 + n * sstep;
#pragma unroll
                for (int g = 0; g < 4; ++g) {
                    __builtin_amdgcn_raw_ptr_buffer_load_lds(rs, dst + g * 4096, 16, lane * 16u, sa + g * 4096u, 0, 17);
                    __builtin_amdgcn_raw_ptr_buffer_load_lds(rs, dst + g * 4096, 16, lane * 16u, sa + g * 4096u, 1024, 17);
                    __builtin_amdgcn_raw_ptr_buffer_load_lds(rs, dst + g * 4096, 16, lane * 16u, sa + g * 4096u, 2048, 17);
                    __builtin_amdgcn_raw_ptr_buffer_load_lds(rs, dst + g * 4096, 16, lane * 16u, sa + g * 4096u, 3072, 17);
                }
                WAIT_VM0();
                CTL[s * 2u + h] = n + 1u;
            }
            if (dead) {
                __hip_atomic_store(tmo, call, __ATOMIC_RELAXED, __HIP_MEMORY_SCOPE_AGENT);
                CTL[8] = 1u; CTL[9] = 1u;
            }
            if (wave < 8 + GAW) {
                const unsigned want = pass ? call : 0u;
                unsigned best = 0u, sbits = 0x80000000u;
                bool sig = false;
                const unsigned own = MALL ? 255u : 1u << xcd;
                u64 v = __hip_atomic_load(gp, __ATOMIC_RELAXED, __HIP_MEMORY_SCOPE_SYSTEM);
                for (unsigned it = 1;; ++it) {
                    const bool ok = (unsigned)(v >> 32) >= want;
                    const unsigned gm = ok ? (unsigned)v | (pass ^ 1u) : 0u;
                    const bool all = __all(ok), newer = __any(gm > best);
                    if (all && !newer) break;
                    if (MPOS == 3 && !sig && (((unsigned)(__ballot(ok) >> (rank * 8u)) & own) == own)) { sig = true; CTL[pass ? 20u : 19u] = 1u; }
                    if (GSLEEP && __builtin_popcountll(__ballot(ok)) < GTHR) __builtin_amdgcn_s_sleep(GSLEEP);
                    v = __hip_atomic_load(gp, __ATOMIC_RELAXED, __HIP_MEMORY_SCOPE_SYSTEM);
                    if (newer) {
                        best = wave_umax(gm);
                        sbits = best == BAD ? BAD : __float_as_uint(127.0f / __uint_as_float(best)) | (best >= 0x7f800000u ? 0x80000000u : 0u);
                    }
                    if (all) break;
                    if ((it & 63u) == 0u) {
                        if (rt() - t0 > DEADLINE_TICKS ||
                            __builtin_amdgcn_readfirstlane(__hip_atomic_load(tmo, __ATOMIC_RELAXED, __HIP_MEMORY_SCOPE_AGENT)) == call) {
                            __hip_atomic_store(tmo, call, __ATOMIC_RELAXED, __HIP_MEMORY_SCOPE_AGENT);
                            sbits = BAD;
                            break;
                        }
                    }
                }
                CTL[pass ? 16u : 19u] = sbits;
            }
            WAIT_LGKM0();
        }
        return;
    }

    const int wm = wave >> 1, wn = wave & 1;
    const int g = lane >> 4, i = lane & 15;
    const unsigned rb = rbp0 * 2u + ((unsigned)wm >> 1);
    const unsigned sidx = rb < 20u ? rb / 5u : 4u + (rb - 20u) / 3u;
    const unsigned sblk = rb < 20u ? rb % 5u : 5u + (rb - 20u) % 3u;
    const unsigned rloc = ((unsigned)wm & 1u) * 32u;
    const unsigned grow = moved ? rank * 512u + (5u + (mem & 1u) + ((unsigned)wm >> 1)) * 64u + rloc
                                : ((sidx + grp * 4u) & 7u) * 512u + sblk * 64u + rloc;
    const unsigned prow = ((sidx + grp * 4u + 4u) & 7u) * 512u + sblk * 64u + rloc;
    const bool remote = cb >= 4u;
    const bool watch = cb0 < 4u && !(OWNROWS && rbp0 == ((0xfdcau >> (mem * 4u)) & 15u));
    const unsigned ocol = cb * 128u + (unsigned)wn * 64u;
    const unsigned msg = (rbp0 * 4u + (remote ? mem : cb)) * 8u + (unsigned)wave;
    const __amdgpu_buffer_rsrc_t rm = mkrsrc(p.wsr[remote ? (cb - mem) & 7u : 0u], WS_USED);
    const __amdgpu_buffer_rsrc_t ro = mkrsrc(p.out, OUT_BYTES);
    GAS u64* g1 = (GAS u64*)(wsme + G1_OFF + xcd * 1024u + (sl * 8u + (unsigned)wave) * 8u);
    asm volatile("" : "+s"(g1));
    if (GWARM && lane == 0) __hip_atomic_store(g1 + (G0_OFF - G1_OFF) / 8, (u64)(call - 1u) << 32, __ATOMIC_RELAXED, __HIP_MEMORY_SCOPE_WORKGROUP);
    if (CPRIO) __builtin_amdgcn_s_setprio(CPRIO);
    if (watch) {
        const v4u mark = {0x80808080u, 0x80808080u, 0x80808080u, 0x80808080u};
        st128(mark, rm, lane * 16u, D_OFF + msg * 2048u);
        st128(mark, rm, lane * 16u + 1024u, D_OFF + msg * 2048u);
    }
    f32x4 acc[2][4];
#pragma unroll
    for (int m = 0; m < 2; ++m)
#pragma unroll
        for (int f = 0; f < 4; ++f) acc[m][f] = (f32x4){0.f, 0.f, 0.f, 0.f};
    const unsigned a_off0 = (unsigned)(32 * wm + i) * 128u + (unsigned)((g ^ (i & 7)) * 16);
    const unsigned a_off1 = (unsigned)(32 * wm + i) * 128u + (unsigned)(((4 + g) ^ (i & 7)) * 16);
    const unsigned b_off = 16384u + (unsigned)(i & 1) * 4096u + (unsigned)(g * 16 + ((8 * wn + (i >> 1)) ^ (8 * (i & 1)))) * 16u;
    bool dead = false;
    unsigned mycnt = 0;
    unsigned last = 3;
    const volatile LAS v2u* fullp = (const volatile LAS v2u*)(L8 + CTL_OFF + (lane < 4 ? lane : 4) * 8);
    WAIT_VM0();
    for (unsigned n = 0; n < (unsigned)KSTEPS; ++n) {
        unsigned m;
        while (true) {
            const v2u f = *fullp;
            m = (unsigned)__ballot(min(f.x, f.y) > mycnt) & 31u;
            if (m) break;
            __builtin_amdgcn_s_sleep(CSLEEP);
        }
        if (m & 16u) { dead = true; break; }
        if (TWARM && n == (unsigned)TWARM)
            asm volatile("buffer_load_dword v104, off, %0, %1 sc0 sc1\n\t"
                         "buffer_load_dword v105, off, %0, %2 sc0 sc1\n\t"
                         "buffer_load_dword v106, off, %3, %4 sc0 sc1\n\t"
                         "buffer_load_dword v107, off, %3, %5 sc0 sc1"
                         :: "s"(rm), "s"(M_OFF + msg * 256u), "s"(D_OFF + msg * 2048u), "s"(ro), "s"(grow * 512u), "s"(prow * 512u)
                         : "v104", "v105", "v106", "v107");
        const unsigned s = (last + 1u + (unsigned)__builtin_ctz((m | (m << 4)) >> (last + 1u))) & 3u;
        last = s;
        asm volatile("" ::: "memory");
        const LAS unsigned char* sp = L8 + s * SLOT_BYTES;
        v4u af[2][2], bf[2][4];
#pragma unroll
        for (int kb = 0; kb < 2; ++kb) {
            af[kb][0] = *(const LAS v4u*)(sp + (kb ? a_off1 : a_off0));
#pragma unroll
            for (int f = 0; f < 4; ++f) bf[kb][f] = *(const LAS v4u*)(sp + b_off + kb * 8192 + f * 1024);
            af[kb][1] = *(const LAS v4u*)(sp + (kb ? a_off1 : a_off0) + 2048);
        }
        mycnt += ((unsigned)lane == s) ? 1u : 0u;
        const unsigned fa = ctl + 48u + s * 4u;
        __builtin_amdgcn_sched_barrier(0);
#pragma unroll
        for (int mm = 0; mm < 2; ++mm)
#pragma unroll
            for (int f = 0; f < 4; ++f)
                acc[mm][f] = __builtin_amdgcn_mfma_f32_16x16x32_bf16(__builtin_bit_cast(bf16x8, af[0][mm]),
                                                                     __builtin_bit_cast(bf16x8, bf[0][f]), acc[mm][f], 0, 0, 0);
        __builtin_amdgcn_sched_barrier(0);
        asm volatile("s_waitcnt lgkmcnt(0)\n\t"
                     "s_mov_b64 exec, 1\n\t"
                     "ds_add_u32 %6, %7\n\t"
                     "s_mov_b64 exec, -1"
                     : "+v"(af[1][0]), "+v"(af[1][1]), "+v"(bf[1][0]), "+v"(bf[1][1]), "+v"(bf[1][2]), "+v"(bf[1][3])
                     : "v"(fa), "v"(1u) : "memory");
        __builtin_amdgcn_sched_barrier(0);
#pragma unroll
        for (int mm = 0; mm < 2; ++mm)
#pragma unroll
            for (int f = 0; f < 4; ++f)
                acc[mm][f] = __builtin_amdgcn_mfma_f32_16x16x32_bf16(__builtin_bit_cast(bf16x8, af[1][mm]),
                                                                     __builtin_bit_cast(bf16x8, bf[1][f]), acc[mm][f], 0, 0, 0);
    }
    TS(ts5);

    {
        float mx = 0.f;
#pragma unroll
        for (int m = 0; m < 2; ++m)
#pragma unroll
            for (int f = 0; f < 4; ++f)
#pragma unroll
                for (int r = 0; r < 4; ++r) mx = fmaxf(mx, acc[m][f][r]);
        const unsigned wmx = dead ? BAD : wave_umax(__float_as_uint(mx));
        if (lane == 0) {
            if (FASTG) __hip_atomic_store(g1 + (G0_OFF - G1_OFF) / 8, ((u64)call << 32) | wmx, __ATOMIC_RELAXED, __HIP_MEMORY_SCOPE_WORKGROUP);
            __hip_atomic_store(g1, ((u64)call << 32) | wmx, __ATOMIC_RELAXED, __HIP_MEMORY_SCOPE_SYSTEM);
        }
    }
    TS(ts6);
    if (CPRIO) __builtin_amdgcn_s_setprio(0);
    asm volatile("" : "+v"(acc[0][0]), "+v"(acc[0][1]), "+v"(acc[0][2]), "+v"(acc[0][3]),
                      "+v"(acc[1][0]), "+v"(acc[1][1]), "+v"(acc[1][2]), "+v"(acc[1][3]) :: "memory");
    const unsigned xb = ((unsigned)wave & 3u) * SLOT_BYTES + ((unsigned)wave >> 2) * 16384u;
    const unsigned ovo = (unsigned)g * 2048u + (unsigned)i * 4u;
    if (remote) {
        unsigned mk = 0;
#pragma unroll
        for (int j = 7; j >= 0; --j)
#pragma unroll
            for (int f = 3; f >= 0; --f) mk = __builtin_amdgcn_alignbit(mk, __float_as_uint(acc[j >> 2][f][j & 3]), 31u);
        mk = ~mk;
        if (mk == 0u) { mk = 1u; acc[0][0][0] = 0.f; }
        if (MPOS == 0) __builtin_amdgcn_raw_buffer_store_b32(mk, rm, lane * 4u, M_OFF + msg * 256u, 17);
        unsigned o[8];
        const unsigned cnt = opaque(positions(mk, o));
        for (unsigned it = 1; !dead && CTL[12 + (wave & 3)] < 128u; ++it)
            if ((it & 63u) == 0u && CTL[8] != 0u) dead = true;
        asm volatile("" ::: "memory");
        LAS f32x4* const xp = (LAS f32x4*)(L8 + xb + 16 + lane * 16);
        for (unsigned q = cnt + (unsigned)lane; q < max(1024u, cnt + 16u); q += 64u) *(LAS float*)(L8 + xb + 16 + q * 4u) = 0.f;
#pragma unroll
        for (int j = 0; j < 8; ++j) {
            unsigned a = xb + 16u + o[j] * 4u;
#pragma unroll
            for (int f = 0; f < 4; ++f) {
                const float v = acc[j >> 2][f][j & 3];
                const unsigned sg = (unsigned)(__float_as_int(v) >> 31);
                asm volatile("ds_write_b32 %0, %1 offset:%2" :: "v"(a | (sg & 0x40000u)), "v"(v), "n"(f * 4) : "memory");
                a += sg << 2;
            }
        }
        if (MPOS == 1) __builtin_amdgcn_raw_buffer_store_b32(mk, rm, lane * 4u, M_OFF + msg * 256u, 17);
        f32x4 d[8];
#pragma unroll
        for (int c = 0; c < 8; ++c) d[c] = xp[c * 64];
        asm volatile("s_waitcnt lgkmcnt(0)" : "+v"(d[0]), "+v"(d[1]), "+v"(d[2]), "+v"(d[3]),
                                                       "+v"(d[4]), "+v"(d[5]), "+v"(d[6]), "+v"(d[7]) :: "memory");
        const unsigned lim = (unsigned)(lane & ~3) * 4u;
        const unsigned dso = opaque(D_OFF + msg * 2048u);
        unsigned sb;
        if (MPOS == 3) {
            while (__builtin_amdgcn_readfirstlane(CTL[20] | CTL[16]) == 0u) { }
            __builtin_amdgcn_raw_buffer_store_b32(mk, rm, lane * 4u, M_OFF + msg * 256u, 17);
        }
        do { sb = __builtin_amdgcn_readfirstlane(CTL[16]); } while (sb == 0u);
        sb = opaque(sb);
        TS(ts7);
        if (sb != BAD) {
            if (MPOS == 2) __builtin_amdgcn_raw_buffer_store_b32(mk, rm, lane * 4u, M_OFF + msg * 256u, 17);
            const float inv = __uint_as_float(sb);
            v4u q;
#pragma unroll
            for (int c = 0; c < 4; ++c) q[c] = quant4(d[c], inv, 12582912.f);
            asm volatile("" : "+v"(q) :: "memory");
            int acked = 0;
            while (acked < wave + 1 - OUT_WIN) acked = (int)__builtin_amdgcn_readfirstlane(CTL[17]);
            st128(q, rm, lane * 16u, dso);
#pragma unroll
            for (int c = 4; c < 8; ++c) {
                if (cnt <= 256u * c) break;
                const unsigned e = quant4(d[c], inv, 12582912.f);
                if (lim + 256u * c < cnt) __builtin_amdgcn_raw_buffer_store_b32(e, rm, lane * 4u + 256u * c, dso, 17);
            }
        }
        asm volatile("s_waitcnt vmcnt(0)\n\t"
                     "s_mov_b64 exec, 1\n\t"
                     "ds_add_u32 %0, %1 offset:68\n\t"
                     "s_mov_b64 exec, -1" :: "v"(ctl), "v"(1u) : "memory");
        return;
    }

#pragma unroll
    for (int m = 0; m < 2; ++m)
#pragma unroll
        for (int f = 0; f < 4; ++f)
#pragma unroll
            for (int r = 0; r < 4; ++r) acc[m][f][r] = __int_as_float(max(__float_as_int(acc[m][f][r]), 0));
    asm volatile("" : "+v"(acc[0][0]), "+v"(acc[0][1]), "+v"(acc[0][2]), "+v"(acc[0][3]),
                      "+v"(acc[1][0]), "+v"(acc[1][1]), "+v"(acc[1][2]), "+v"(acc[1][3]) :: "memory");
    unsigned sb;
    do { sb = __builtin_amdgcn_readfirstlane(CTL[16]); } while (sb == 0u);
    TS(ts7);
    dead = sb == BAD;
    unsigned mk = __builtin_amdgcn_raw_buffer_load_b32(rm, lane * 4u, M_OFF + msg * 256u, 17);
    {
        const float inv = dead ? 0.f : __uint_as_float(sb);
        const float magic = dead ? 12583040.f : 12582912.f;
#pragma unroll
        for (int m = 0; m < 2; ++m) {
            const f32x2 iv = {inv, inv}, mg = {magic, magic};
            f32x2 z[4][2];
#pragma unroll
            for (int f = 0; f < 4; ++f) {
                z[f][0] = __builtin_elementwise_fma((f32x2){acc[m][f][0], acc[m][f][1]}, iv, mg);
                z[f][1] = __builtin_elementwise_fma((f32x2){acc[m][f][2], acc[m][f][3]}, iv, mg);
            }
#pragma unroll
            for (int r = 0; r < 4; ++r) {
                const unsigned lo = __builtin_amdgcn_perm(__float_as_uint(z[1][r >> 1][r & 1]), __float_as_uint(z[0][r >> 1][r & 1]), 0x0c0c0400u);
                const unsigned hi = __builtin_amdgcn_perm(__float_as_uint(z[3][r >> 1][r & 1]), __float_as_uint(z[2][r >> 1][r & 1]), 0x04000c0cu);
                __builtin_amdgcn_raw_buffer_store_b32(lo | hi, ro, ovo + r * 512u, (grow + 16u * m) * 512u + ocol, OAUX);
            }
        }
    }
    if (watch && !dead) {
        unsigned ra[8], sel[8], o[8];
        bool late = false;
        for (unsigned it = 1; __any(mk == 0u); ++it) {
            mk = __builtin_amdgcn_raw_buffer_load_b32(rm, lane * 4u, M_OFF + msg * 256u, 17);
            if ((it & 63u) == 0u && (rt() - t0 > DEADLINE_TICKS ||
                __builtin_amdgcn_readfirstlane(__hip_atomic_load(tmo, __ATOMIC_RELAXED, __HIP_MEMORY_SCOPE_AGENT)) == call)) { late = true; break; }
        }
        __builtin_amdgcn_raw_buffer_store_b32(0u, rm, lane * 4u, M_OFF + msg * 256u, 17);
        const unsigned cnt = positions(mk, o);
#pragma unroll
        for (int j = 0; j < 8; ++j) {
            const unsigned ps = (((mk >> (4 * j)) & 15u) * 0x00204081u) & 0x01010101u;
            unsigned rk = (ps << 8) | (o[j] & 3u);
            rk += rk << 8;
            asm("" : "+v"(rk));
            rk += rk << 16;
            const unsigned am = ps + 0x0f0f0f0fu;
            sel[j] = (rk & ~am) | (0x0c0c0c0cu & am);
            ra[j] = xb + (o[j] & ~3u);
        }
        unsigned vo1 = (unsigned)lane * 16u + 1024u < cnt ? (unsigned)lane * 16u + 1024u : 0x80000000u;
        asm volatile("" : "+v"(ra[0]), "+v"(ra[1]), "+v"(ra[2]), "+v"(ra[3]), "+v"(ra[4]), "+v"(ra[5]), "+v"(ra[6]), "+v"(ra[7]),
                          "+v"(sel[0]), "+v"(sel[1]), "+v"(sel[2]), "+v"(sel[3]), "+v"(sel[4]), "+v"(sel[5]), "+v"(sel[6]), "+v"(sel[7]), "+v"(vo1) :: "memory");
        const unsigned pso = opaque(prow * 512u + ocol);
        while (!late) {
            unsigned got, t, t2;
#define DLD(a, b) "buffer_load_dwordx4 v[" #a "], %3, %5, %6 offen sc0 sc1\n\tbuffer_load_dwordx4 v[" #b "], %4, %5, %6 offen sc0 sc1\n\ts_sleep %9\n\t"
#define DCK(k, r0, r1, r2, r3, r4, r5, r6, r7) "s_waitcnt vmcnt(%10)\n\t" \
            "v_or3_b32 %1, v" #r0 ", v" #r1 ", v" #r2 "\n\tv_or3_b32 %2, v" #r3 ", v" #r4 ", v" #r5 "\n\tv_or3_b32 %1, %1, v" #r6 ", v" #r7 "\n\t" \
            "v_or_b32 %1, %1, %2\n\tv_and_b32 %1, 0x80808080, %1\n\tv_cmp_ne_u32 vcc, 0, %1\n\ts_cbranch_vccz .Ldg" #k "%=\n\t" DLD(r0:r3, r4:r7)
#define DWR(k, r0, r1, r2, r3, r4, r7) ".Ldg" #k "%=:\n\tds_write2st64_b32 %7, v" #r0 ", v" #r1 " offset1:1\n\tds_write2st64_b32 %7, v" #r2 ", v" #r3 " offset0:2 offset1:3\n\t" \
            "ds_write_b128 %8, v[" #r4 ":" #r7 "] offset:1024\n\ts_mov_b32 %0, 1\n\t"
            asm volatile(DLD(104:107, 108:111)
                         "s_movk_i32 %0, 100\n"
                         ".Ldl%=:\n\t"
                         DCK(0, 104, 105, 106, 107, 108, 109, 110, 111)
                         "s_sub_u32 %0, %0, 1\n\t"
                         "s_cbranch_scc0 .Ldl%=\n\t"
                         "s_mov_b32 %0, 0\n\t"
                         "s_branch .Lde%=\n"
                         DWR(0, 104, 105, 106, 107, 108, 111)
                         "\n.Lde%=:"
                         : "=&s"(got), "=&v"(t), "=&v"(t2)
                         : "v"(lane * 16u), "v"(vo1), "s"(rm), "s"(D_OFF + msg * 2048u), "v"(xb + lane * 4u), "v"(xb + lane * 16u), "n"(DPD > 1 ? DPS : 0), "n"(2 * DPD - 2)
                         : "vcc", "scc", "memory", "v104", "v105", "v106", "v107", "v108", "v109", "v110", "v111", "v112", "v113", "v114", "v115",
                           "v116", "v117", "v118", "v119", "v120", "v121", "v122", "v123", "v124", "v125", "v126", "v127");
            if (got) break;
            if (rt() - t0 > DEADLINE_TICKS ||
                __builtin_amdgcn_readfirstlane(__hip_atomic_load(tmo, __ATOMIC_RELAXED, __HIP_MEMORY_SCOPE_AGENT)) == call) late = true;
        }
        if (late) {
            __hip_atomic_store(tmo, call, __ATOMIC_RELAXED, __HIP_MEMORY_SCOPE_AGENT);
            __builtin_amdgcn_raw_buffer_store_b32(0x80808080u, ro, ovo, prow * 512u + ocol, 17);
        } else {
            v2u e[8];
#pragma unroll
            for (int j = 0; j < 8; ++j) {
                const LAS unsigned* rp = (const LAS unsigned*)(L8 + ra[j]);
                e[j] = (v2u){rp[0], rp[1]};
            }
            __builtin_amdgcn_sched_barrier(0);
#pragma unroll
            for (int j = 0; j < 8; ++j)
                __builtin_amdgcn_raw_buffer_store_b32(__builtin_amdgcn_perm(e[j].y, e[j].x, sel[j]), ro, ovo + (j & 3) * 512u, pso + (j >> 2) * 8192u, OAUX);
        }
    }
}

extern "C" void kernel_launch(int rank, int nranks, ncclComm_t comm, hipStream_t stream,
                              void* const* d_in, const int* in_sizes, int n_in,
                              void* d_out, int out_size, const int* in_dtypes, int out_dtype,
                              void* const* d_out_all, void* const* d_ws_all, size_t ws_bytes) {
    if (nranks != NR || n_in < 2 || ws_bytes < (size_t)WS_USED + 4096u || (unsigned)rank >= (unsigned)NR ||
        in_sizes[0] != 512 * 4096 || in_sizes[1] != 4096 * 512 || out_size != 4096 * 512) {
        fprintf(stderr, "unexpected problem configuration\n");
        return;
    }
    Params p;
    p.x = d_in[0];
    p.w = d_in[1];
    for (int d = 0; d < NR; ++d) p.wsr[d] = (char*)d_ws_all[(rank + d) & 7];
    p.out = d_out;
    p.rank = rank;
    ag_gemm_kernel<<<dim3(NWG), dim3(1024), 0, stream>>>(p);
}
```

```cpp
#include <hip/hip_runtime.h>
#include <rccl/rccl.h>
#include <cstdio>
#include <cstdint>

#define LAS __attribute__((address_space(3)))
#define GAS __attribute__((address_space(1)))
typedef unsigned v4u __attribute__((ext_vector_type(4)));
typedef unsigned v2u __attribute__((ext_vector_type(2)));
typedef float f32x4 __attribute__((ext_vector_type(4)));
typedef float f32x2 __attribute__((ext_vector_type(2)));
typedef __bf16 bf16x8 __attribute__((ext_vector_type(8)));
typedef unsigned long long u64;

#ifndef CU_PER_DEST
#define CU_PER_DEST 8
#endif
#define WV_PER_CU 3
#define WIN 6
#define SELF_CU 16
#define SELF_WV 4
#define OUT_WIN 4
#define XCOPY 1
#define WLIN 1
#define FASTG 1
#define COVL 1
#define WARM 1
#define AP 4
#define CSLEEP 1
#define LSLEEP 1
#define GWARM 1
#define CPRIO 0
#define AGW 1
#define GAW 1
#define FPD 1
#define FPS 4
#define FPN 15
#define DPD 1
#define DPS 4
#define MPOS 3
#define FDELAY 0
#define FSLEEP 0
#define GSLEEP 16
#define GTHR 48
#define ASLEEP 0
#define TWARM 0
#define MALL 0
#define AAUX 17
#define OAUX 17
#define SI 4
#define ZSLEEP 127
#define RSLEEP 0
#define OLDF 1
#define PIPE 1
#define TN 2
#define EAGER 4
#define PF 0
#define OWNROWS 1

constexpr int NR = 8;
constexpr int NWG = 256;
constexpr int NGW = 128;
constexpr int KSTEPS = 64;
constexpr int NSLOT = 4;
constexpr unsigned NRB = 36;
constexpr unsigned A_OFF = 0;
constexpr unsigned B_OFF = 18u << 20;
constexpr unsigned F_OFF = 26u << 20;
constexpr unsigned FA_OFF = F_OFF;
constexpr unsigned FB_OFF = F_OFF + 12288;
constexpr unsigned Q_OFF = F_OFF + 16384;
constexpr unsigned TM_OFF = F_OFF + 18432;
constexpr unsigned R_OFF = F_OFF + 20480;
constexpr unsigned G1_OFF = F_OFF + 24576;
constexpr unsigned CT_OFF = F_OFF + 32768;
constexpr unsigned G0_OFF = F_OFF + 49152;
constexpr unsigned G2_OFF = F_OFF + 65536;
constexpr unsigned DBG_OFF = F_OFF + 131072;
constexpr unsigned M_OFF = F_OFF + (512u << 10);
constexpr unsigned D_OFF = 27u << 20;
constexpr unsigned WS_USED = 28u << 20;
constexpr unsigned X_BYTES = 512u * 4096u * 2u;
constexpr unsigned OUT_BYTES = 4096u * 512u;
constexpr int SLOT_BYTES = 32768;
constexpr int CTL_OFF = NSLOT * SLOT_BYTES;
constexpr int LDS_BYTES = CTL_OFF + 128;
constexpr u64 DEADLINE_TICKS = 40000000ull;
constexpr int RSRC3 = 0x00020000;
constexpr unsigned BAD = 0xffffffffu;
static_assert(LDS_BYTES <= 160 * 1024, "LDS");

struct Params {
    const void* x;
    const void* w;
    char* wsr[NR];
    void* out;
    int rank;
};

__device__ __forceinline__ unsigned opaque(unsigned v) { asm volatile("" : "+s"(v)); return v; }
__device__ __forceinline__ u64 rt() { return __builtin_amdgcn_s_memrealtime(); }
__device__ __forceinline__ __amdgpu_buffer_rsrc_t mkrsrc(const void* p, unsigned bytes) {
    return __builtin_amdgcn_make_buffer_rsrc((void*)p, (short)0, (int)bytes, RSRC3);
}
#define WAIT_VM0() asm volatile("s_waitcnt vmcnt(0)" ::: "memory")
#define WAIT_VM0_B() do { asm volatile("" ::: "memory"); __builtin_amdgcn_s_waitcnt(0x0F70); asm volatile("" ::: "memory"); } while (0)
#define WAIT_LGKM0() asm volatile("s_waitcnt lgkmcnt(0)" ::: "memory")

__device__ __forceinline__ void st128(const v4u d, const __amdgpu_buffer_rsrc_t r, unsigned vo, unsigned so) {
    __builtin_amdgcn_raw_buffer_store_b128(d, r, vo, so, 17);
    asm volatile("s_nop 1" :: "v"(d) : "memory");
}

#define STAMPV(idx, val) do { } while (0)
#define STAMP(idx) do { } while (0)
#define TS(name) do { } while (0)

__device__ __forceinline__ unsigned wave_umax(unsigned v) {
    v = max(v, (unsigned)__builtin_amdgcn_update_dpp(0, (int)v, 0x111, 0xf, 0xf, true));
    v = max(v, (unsigned)__builtin_amdgcn_update_dpp(0, (int)v, 0x112, 0xf, 0xf, true));
    v = max(v, (unsigned)__builtin_amdgcn_update_dpp(0, (int)v, 0x114, 0xf, 0xf, true));
    v = max(v, (unsigned)__builtin_amdgcn_update_dpp(0, (int)v, 0x118, 0xf, 0xf, true));
    const unsigned a = __builtin_amdgcn_readlane(v, 15), b = __builtin_amdgcn_readlane(v, 31);
    const unsigned c = __builtin_amdgcn_readlane(v, 47), d = __builtin_amdgcn_readlane(v, 63);
    return max(max(a, b), max(c, d));
}

__device__ __forceinline__ unsigned wave_scan(unsigned v) {
    v += (unsigned)__builtin_amdgcn_update_dpp(0, (int)v, 0x111, 0xf, 0xf, true);
    v += (unsigned)__builtin_amdgcn_update_dpp(0, (int)v, 0x112, 0xf, 0xf, true);
    v += (unsigned)__builtin_amdgcn_update_dpp(0, (int)v, 0x114, 0xf, 0xf, true);
    v += (unsigned)__builtin_amdgcn_update_dpp(0, (int)v, 0x118, 0xf, 0xf, true);
    v += (unsigned)__builtin_amdgcn_update_dpp(0, (int)v, 0x142, 0xa, 0xf, false);
    v += (unsigned)__builtin_amdgcn_update_dpp(0, (int)v, 0x143, 0xc, 0xf, false);
    return v;
}

__device__ __forceinline__ unsigned positions(unsigned mk, unsigned (&o)[8]) {
    unsigned x = mk;
    x -= (x >> 1) & 0x55555555u;
    x = (x & 0x33333333u) + ((x >> 2) & 0x33333333u);
    unsigned base = 0;
    unsigned c[2], e[2], cl[2], el[2];
    c[0] = x & 0x0f0f0f0fu;
    c[1] = (x >> 4) & 0x0f0f0f0fu;
#pragma unroll
    for (int h = 0; h < 2; ++h) {
        e[h] = wave_scan(c[h]) - c[h];
        cl[h] = __builtin_amdgcn_readlane(c[h], 63);
        el[h] = __builtin_amdgcn_readlane(e[h], 63);
    }
#pragma unroll
    for (int j = 0; j < 8; ++j) {
        o[j] = base + ((e[j & 1] >> (8 * (j >> 1))) & 255u);
        base += ((cl[j & 1] >> (8 * (j >> 1))) & 255u) + ((el[j & 1] >> (8 * (j >> 1))) & 255u);
    }
    return base;
}

__device__ __forceinline__ unsigned quant4(const f32x4 d, float inv, float magic) {
    const f32x2 iv = {inv, inv}, mg = {magic, magic};
    const f32x2 a = __builtin_elementwise_fma((f32x2){d[0], d[1]}, iv, mg), b = __builtin_elementwise_fma((f32x2){d[2], d[3]}, iv, mg);
    return __builtin_amdgcn_perm(__float_as_uint(a[1]), __float_as_uint(a[0]), 0x0c0c0400u) |
           __builtin_amdgcn_perm(__float_as_uint(b[1]), __float_as_uint(b[0]), 0x04000c0cu);
}

__global__ void __launch_bounds__(1024, 1) ag_gemm_kernel(Params p) {
    __shared__ __attribute__((aligned(16))) unsigned char lds[LDS_BYTES];
    LAS unsigned char* L8 = (LAS unsigned char*)lds;
    volatile LAS unsigned* CTL = (volatile LAS unsigned*)(L8 + CTL_OFF);
    const int tid = threadIdx.x;
    const int wave = __builtin_amdgcn_readfirstlane(tid >> 6);
    const int lane = tid & 63;
    const unsigned rank = (unsigned)p.rank & 7u;
    const unsigned grp = rank >> 2, mem = rank & 3u;
    const unsigned b = blockIdx.x;
    TS(tsk);
    char* const wsme = p.wsr[0];
    GAS unsigned* ctr = (GAS unsigned*)(wsme + CT_OFF + (b * 16 + wave) * 4);
    GAS unsigned* tmo = (GAS unsigned*)(wsme + TM_OFF);
    const unsigned ctl = (unsigned)CTL_OFF;

    if (b < (unsigned)(NWG - NGW)) {
        if (b - 56u < 8u) {
            if (wave >= AGW) return;
            const u64 t0 = rt();
            const unsigned j = b - 56u;
            const unsigned call = __builtin_amdgcn_readfirstlane(__hip_atomic_load(ctr, __ATOMIC_RELAXED, __HIP_MEMORY_SCOPE_AGENT)) + 1u;
            if (lane == 0) __hip_atomic_store(ctr, call, __ATOMIC_RELAXED, __HIP_MEMORY_SCOPE_AGENT);
            GAS u64* const dstp = (GAS u64*)(p.wsr[lane & 7] + G2_OFF + (unsigned)(lane >> 3) * 8192u + (rank * 8u + j) * 128u);
            const __amdgpu_buffer_rsrc_t rs = mkrsrc(wsme, WS_USED);
#pragma nounroll
            for (unsigned pass = opaque(WARM ? 0u : 1u); pass < 2u; pass = opaque(pass + 1u)) {
                const unsigned want = pass ? call : 0u;
                unsigned gm;
                for (unsigned it = 1;; ++it) {
                    const v4u v = __builtin_amdgcn_raw_buffer_load_b128(rs, lane * 16u, (FASTG && (it & 255u) ? G0_OFF : G1_OFF) + j * 1024u, AAUX);
                    gm = max(v.x, v.z);
                    if (__all(v.y >= want && v.w >= want)) break;
                    if (ASLEEP) __builtin_amdgcn_s_sleep(ASLEEP);
                    if ((it & 255u) == 0u) {
                        if (rt() - t0 > DEADLINE_TICKS ||
                            __builtin_amdgcn_readfirstlane(__hip_atomic_load(tmo, __ATOMIC_RELAXED, __HIP_MEMORY_SCOPE_AGENT)) == call) {
                            __hip_atomic_store(tmo, call, __ATOMIC_RELAXED, __HIP_MEMORY_SCOPE_AGENT);
                            gm = BAD;
                            break;
                        }
                    }
                }
                gm = wave_umax(gm);
                if (XCOPY || lane < 8) __hip_atomic_store(dstp + (pass ? 0 : 8), ((u64)call << 32) | gm, __ATOMIC_RELAXED, __HIP_MEMORY_SCOPE_SYSTEM);
            }
            STAMP(9 + wave);
            return;
        }
        if (b - 81u < 7u) {
            const unsigned d = b - 80u, src = (rank + d) & 7u;
            const unsigned call_v = __hip_atomic_load(ctr, __ATOMIC_RELAXED, __HIP_MEMORY_SCOPE_AGENT) + 1u;
            const __amdgpu_buffer_rsrc_t rs = mkrsrc(wsme, WS_USED);
            const bool same = (src >> 2) == grp;
            if (ZSLEEP) __builtin_amdgcn_s_sleep(ZSLEEP);
            const v4u z = {0u, 0u, 0u, 0u};
            for (unsigned q = (unsigned)wave; q < 160u * TN; q += 16u) {
                const unsigned ks = 64u - 4u * TN + q / 40u, r = q % 40u;
                const unsigned so = same ? A_OFF + (ks * NRB + (src & 3u) * 5u) * 8192u + r * 1024u
                                  : r < 24u ? A_OFF + (ks * NRB + 20u + (src & 3u) * 3u) * 8192u + r * 1024u
                                            : B_OFF + (ks * 8u + 4u + (src & 3u)) * 16384u + (r - 24u) * 1024u;
                st128(z, rs, lane * 16u, so);
            }
            WAIT_VM0_B();
            __builtin_amdgcn_s_barrier();
            if (tid == 0) {
                __hip_atomic_store(ctr, call_v, __ATOMIC_RELAXED, __HIP_MEMORY_SCOPE_AGENT);
                __hip_atomic_store((GAS unsigned*)(p.wsr[d] + R_OFF + (8u - d) * 128u), call_v, __ATOMIC_RELAXED, __HIP_MEMORY_SCOPE_SYSTEM);
            }
            return;
        }
        const bool self = b >= 64u;
#if CU_PER_DEST == 8
        const unsigned dsel = self ? 0u : 1u + (b >> 3);
        const unsigned sub = self ? b - 64u : (b & 7u);
#else
        const unsigned dsel = self ? 0u : 1u + 2u * (b >> 3) + (b & 1u);
        const unsigned sub = self ? b - 64u : ((b & 7u) >> 1);
#endif
        if (self ? (sub >= (unsigned)SELF_CU || wave >= SELF_WV) : (dsel >= 8u || wave >= WV_PER_CU)) return;
        const unsigned ncu = self ? (unsigned)SELF_CU : (unsigned)CU_PER_DEST;
        const unsigned nwv = self ? (unsigned)(SELF_CU * SELF_WV) : (unsigned)(CU_PER_DEST * WV_PER_CU);
        const unsigned wv = (unsigned)wave * ncu + sub;
        const unsigned dst = (rank + dsel) & 7u;
        const bool same = (dst >> 2) == grp;
        const unsigned nper = self ? 16u : 5u;
        const unsigned kmul = self ? 4096u : 13108u;
        const unsigned nx = self ? 8u : same ? 5u : 3u;
        const unsigned xb0 = same ? 0u : 5u;
        const unsigned drb0 = same ? mem * 5u : 20u + mem * 3u;
        const unsigned nunits = nper * KSTEPS;
        char* const wsd = p.wsr[dsel];
        const __amdgpu_buffer_rsrc_t rd = mkrsrc(wsd, WS_USED);
        const __amdgpu_buffer_rsrc_t rx = mkrsrc(p.x, X_BYTES), rw = mkrsrc(p.w, X_BYTES);
        const unsigned xoff = (unsigned)(lane >> 3) * 8192u + (unsigned)((lane & 7) ^ (lane >> 3)) * 16u;
        const unsigned q = lane >> 4, i = lane & 15;
        const unsigned woff = q * 8192u + i * 16u;
        const unsigned wst = (i >> 3) * 4096u + (q * 16u + (i & 7u)) * 16u;
        GAS unsigned* qhead = (GAS unsigned*)(wsme + Q_OFF + dsel * 128u);
        v4u cur[8];
        auto load_unit = [&](unsigned u) {
            const unsigned ks = (u * kmul) >> 16, idx = u - ks * nper;
            if (idx < nx) {
                const unsigned so = (xb0 + idx) * (64u * 8192u) + ks * 128u;
#pragma unroll
                for (int t = 0; t < 8; ++t) cur[t] = __builtin_amdgcn_raw_buffer_load_b128(rx, xoff, so + t * 65536u, 0);
            } else {
                const unsigned wi = idx - nx;
                const unsigned so = (ks * 64u + (wi & 1u) * 32u) * 1024u + (self ? (wi >> 1) : (dst & 3u)) * 256u;
#pragma unroll
                for (int t = 0; t < 8; ++t) cur[t] = __builtin_amdgcn_raw_buffer_load_b128(rw, woff + (t & 3) * 1024u, so + (t >> 2) * 4096u, 0);
            }
        };
        load_unit(wv);
        const unsigned call_v = __hip_atomic_load(ctr, __ATOMIC_RELAXED, __HIP_MEMORY_SCOPE_AGENT) + 1u;
        if (tid < 2) CTL[tid] = 0u;
        WAIT_LGKM0();
        __builtin_amdgcn_s_barrier();
        WAIT_VM0_B();
        const unsigned call = __builtin_amdgcn_readfirstlane(call_v);
        const unsigned qbase = (call - 1u) * nunits - nwv;
        const int wbias = self ? (1 << 24) : WIN - 1;
        TS(ts0);
        unsigned u = wv, skip = 0u;
        const GAS unsigned* rdy = (const GAS unsigned*)(wsme + R_OFF + dsel * 128u);
        while (true) {
            const unsigned ks = (u * kmul) >> 16, idx = u - ks * nper;
            const bool isx = idx < nx;
            const unsigned drb = idx < 5u ? drb0 + idx : (mem & 1u) ? (idx == 5u ? 34u : 26u + idx) : 27u + idx;
            const unsigned wi = idx - nx;
            const unsigned dcb = self ? (wi >> 1) : 4u + mem, kb = wi & 1u;
            unsigned seq;
            asm volatile("s_mov_b64 exec, 1\n\t"
                         "ds_add_rtn_u32 %0, %1, %2\n\t"
                         "s_mov_b64 exec, -1"
                         : "=&v"(seq) : "v"(ctl), "v"(1u) : "memory");
            v4u o[8];
            if (isx) {
#pragma unroll
                for (int t = 0; t < 8; ++t) o[t] = cur[t];
            } else {
#pragma unroll
                for (int d = 0; d < 4; ++d) {
#pragma unroll
                    for (int e = 0; e < 4; ++e) {
                        o[2 * d][e] = __builtin_amdgcn_perm(cur[2 * e + 1][d], cur[2 * e][d], 0x05040100u);
                        o[2 * d + 1][e] = __builtin_amdgcn_perm(cur[2 * e + 1][d], cur[2 * e][d], 0x07060302u);
                    }
                }
            }
            const unsigned so = isx ? A_OFF + (ks * NRB + drb) * 8192u : B_OFF + (ks * 8u + dcb) * 16384u + kb * 8192u;
            const unsigned so2 = so + 4096u;
            const unsigned vo = (unsigned)lane * 16u | skip, vo2 = isx ? vo : vo ^ 128u;
            const unsigned fpos = (ks & 3u) * 128u + (ks >> 2) * 8u;
            const unsigned foff = isx ? FA_OFF + (drb >> 1) * 512u + fpos + (drb & 1u) * 4u : FB_OFF + dcb * 512u + fpos + kb * 4u;
            asm volatile("s_waitcnt lgkmcnt(0)" : "+v"(seq) :: "memory");
            const int need = 8 * (int)__builtin_amdgcn_readfirstlane(seq) - wbias;
            int acked = 0;
            unsigned tk;
#pragma unroll
            for (int t = 0; t < 8; ++t) {
                if (t < WIN) {
                    while (acked < need + t) acked = (int)__builtin_amdgcn_readfirstlane(CTL[1]);
                } else {
                    asm volatile("s_waitcnt vmcnt(%2)\n\t"
                                 "s_mov_b64 exec, 1\n\t"
                                 "ds_add_u32 %0, %1 offset:4\n\t"
                                 "s_mov_b64 exec, -1" :: "v"(ctl), "v"(1u), "n"(WIN - 1 + (AP > t - WIN && AP < t ? 1 : 0)) : "memory");
                }
                if (t == AP)
                    asm volatile("s_mov_b64 exec, 1\n\t"
                                 "global_atomic_add %0, %1, %2, %3 sc0\n\t"
                                 "s_mov_b64 exec, -1"
                                 : "=&v"(tk) : "v"(0u), "v"(1u), "s"(qhead) : "memory");
                st128(o[t], rd, (t < 4 ? vo : vo2) + (t & 3) * 1024u, t < 4 ? so : so2);
            }
#define ACK(j) ".if %6 >= 8 - " #j "\n\t.if %7 > " #j "\n\ts_waitcnt vmcnt(8 - " #j ")\n\t.else\n\ts_waitcnt vmcnt(7 - " #j ")\n\t.endif\n\tds_add_u32 %5, %1 offset:4\n\t.endif\n\t"
            asm volatile("s_mov_b64 exec, 1\n\t"
                         ACK(0) ACK(1) ACK(2) ACK(3) ACK(4) ACK(5) ACK(6)
                         "s_waitcnt vmcnt(0)\n\t"
                         "global_store_dword %2, %3, %4 sc0 sc1\n\t"
                         "ds_add_u32 %5, %1 offset:4\n\t"
                         "s_mov_b64 exec, -1"
                         : "+v"(tk) : "v"(1u), "v"(foff), "v"(skip ? call - 1u : call), "s"(wsd), "v"(ctl), "n"(WIN), "n"(AP) : "memory");
            u = __builtin_amdgcn_readfirstlane(tk) - qbase;
            if (u >= nunits) break;
            load_unit(u);
            if (!self && u >= 320u - 20u * TN) {
                const u64 t0 = rt();
                bool late = false;
                while (__builtin_amdgcn_readfirstlane(__hip_atomic_load(rdy, __ATOMIC_RELAXED, __HIP_MEMORY_SCOPE_SYSTEM)) < call)
                    if (rt() - t0 > DEADLINE_TICKS) { late = true; break; }
                if (late) __hip_atomic_store(tmo, call, __ATOMIC_RELAXED, __HIP_MEMORY_SCOPE_AGENT);
                skip = late ? 0x80000000u : 0u;
            }
            WAIT_VM0_B();
        }
        if (lane == 0) __hip_atomic_store(ctr, call, __ATOMIC_RELAXED, __HIP_MEMORY_SCOPE_AGENT);
        STAMP(16 + wave);
        if (wave == 0) { STAMPV(0, ts0); STAMPV(1, tsk); }
        return;
    }

    const u64 t0 = rt();
    const unsigned call_v = __hip_atomic_load(ctr, __ATOMIC_RELAXED, __HIP_MEMORY_SCOPE_AGENT) + 1u;
    if (tid < 32) CTL[tid] = tid == 18 ? BAD : 0u;
    const unsigned gb = b - (unsigned)(NWG - NGW);
    const unsigned xcd = gb & 7u, sl = gb >> 3;
    const unsigned rbp0 = (xcd >> 1) * 4u + (sl & 3u);
    const unsigned cbi = sl >> 2;
    const unsigned cb0 = (cbi & 1u) + 2u * (xcd & 1u) + 4u * (cbi >> 1);
    const bool moved = OWNROWS && cb0 >= 4u && rbp0 == ((0xfdcau >> ((cb0 - 4u) * 4u)) & 15u);
    const unsigned rbp = moved ? 16u : rbp0;
    const unsigned cb = moved ? cb0 - 4u : cb0;
    WAIT_LGKM0();
    __builtin_amdgcn_s_barrier();
    const unsigned call = __builtin_amdgcn_readfirstlane(call_v);
    if (lane == 0) __hip_atomic_store(ctr, call, __ATOMIC_RELAXED, __HIP_MEMORY_SCOPE_AGENT);

    if (wave >= 8) {
        __builtin_amdgcn_s_setprio(2);
        const unsigned s = (unsigned)wave & 3u, h = ((unsigned)wave >> 2) & 1u;
        const __amdgpu_buffer_rsrc_t rs = mkrsrc(wsme, WS_USED);
        const GAS u64* fp = (const GAS u64*)(wsme + (h ? FB_OFF + cb * 512u : FA_OFF + rbp * 512u) + s * 128u + (lane & 15) * 8u);
        const unsigned src0 = h ? B_OFF + (s * 8u + cb) * 16384u : A_OFF + (s * NRB + rbp * 2u) * 8192u;
        const unsigned sstep = h ? 4u * 8u * 16384u : 4u * NRB * 8192u;
        LAS unsigned char* dst = L8 + s * SLOT_BYTES + h * 16384u;
        const GAS u64* gp = (const GAS u64*)(wsme + G2_OFF + (XCOPY ? xcd * 8192u : 0u) + (unsigned)lane * 128u);
        const unsigned rem0 = (h ? cb >= 4u : rbp < 16u && rbp * 2u - mem * 5u >= 5u) ? 1u << 16 : 0u;
        const unsigned rem1 = (h ? cb >= 4u : rbp < 16u && rbp * 2u + 1u - mem * 5u >= 5u) ? 1u << 17 : 0u;
        const GAS u64* sp = lane < 16 ? fp : (const GAS u64*)(wsme + src0 + (16u - TN) * sstep + (lane & 1) * 8192u + SI * 1024u);
        const unsigned spd = lane < 16 ? 0u : sstep;
        unsigned thr = lane < 16 ? call : 1u;
        unsigned known = 0;
        bool dead = false;
        if (wave == 8) STAMP(2);
#pragma nounroll
        for (unsigned pass = opaque(WARM ? 0u : 1u); pass < 2u; pass = opaque(pass + 1u)) {
            const GAS u64* pp = fp;
            thr = lane < 16 ? call : 1u;
            for (unsigned n = pass ? 0u : 15u; n < 16u; ++n) {
                unsigned m0 = 1u << n, m1 = m0;
                if (SI < 8 && n >= 16u - TN && pass) { pp = sp; sp = (const GAS u64*)((const GAS char*)sp + spd); m0 |= rem0; m1 |= rem1; }
                if (EAGER < 4 && n == 15u && s >= (unsigned)EAGER && lane >= 16) thr = 0u;
                bool go = !pass || (known & m0) != 0u;
                for (unsigned it = 1; !dead && !go; ++it) {
                    const u64 f = __hip_atomic_load(pp, __ATOMIC_RELAXED, __HIP_MEMORY_SCOPE_SYSTEM);
                    const unsigned b0 = (unsigned)__ballot((unsigned)f >= thr), b1 = (unsigned)__ballot((unsigned)(f >> 32) >= thr);
                    known = b0 & b1 & 0xffffu;
                    go = (b0 & m0) != 0u && (b1 & m1) != 0u;
                    if ((it & 255u) == 0u && (rt() - t0 > DEADLINE_TICKS || CTL[8] != 0u)) { dead = true; break; }
                }
                for (unsigned it = 1; !dead && CTL[12 + s] < 8u * n * pass; ++it) {
                    if ((it & 63u) == 0u && CTL[8] != 0u) dead = true;
                    __builtin_amdgcn_s_sleep(LSLEEP);
                }
                if (dead) break;
                asm volatile("" ::: "memory");
                const unsigned sa = src0 + n * sstep, lvo = pass ? (unsigned)lane * 16u : 0x80000000u;
                while (true) {
#pragma unroll
                    for (int g = 0; g < 4; ++g) {
                        __builtin_amdgcn_raw_ptr_buffer_load_lds(rs, dst + g * 4096, 16, lvo, sa + g * 4096u, 0, 17);
                        __builtin_amdgcn_raw_ptr_buffer_load_lds(rs, dst + g * 4096, 16, lvo, sa + g * 4096u, 1024, 17);
                        __builtin_amdgcn_raw_ptr_buffer_load_lds(rs, dst + g * 4096, 16, lvo, sa + g * 4096u, 2048, 17);
                        __builtin_amdgcn_raw_ptr_buffer_load_lds(rs, dst + g * 4096, 16, lvo, sa + g * 4096u, 3072, 17);
                    }
                    if ((known >> n) & 1u) { WAIT_VM0(); break; }
                    const u64 f = __hip_atomic_load(fp, __ATOMIC_RELAXED, __HIP_MEMORY_SCOPE_SYSTEM);
                    WAIT_VM0();
                    unsigned c[16];
#pragma unroll
                    for (int g = 0; g < 16; ++g) c[g] = *(const volatile LAS unsigned*)(dst + g * 1024 + lane * 16);
#pragma unroll
                    for (int g = 1; g < 16; ++g) c[0] = min(c[0], c[g]);
                    if (!pass || __all(c[0] != 0u)) break;
                    known = (unsigned)__ballot((unsigned)f >= call && (unsigned)(f >> 32) >= call) & 0xffffu;
                    if (rt() - t0 > DEADLINE_TICKS || CTL[8] != 0u) { dead = true; break; }
                    if (RSLEEP) __builtin_amdgcn_s_sleep(RSLEEP);
                }
                if (dead) break;
                CTL[pass ? s * 2u + h : 19u] = n + 1u;
            }
            if (dead) {
                __hip_atomic_store(tmo, call, __ATOMIC_RELAXED, __HIP_MEMORY_SCOPE_AGENT);
                CTL[8] = 1u; CTL[9] = 1u;
            }
            if (wave < 8 + GAW) {
                const unsigned want = pass ? call : 0u;
                unsigned best = 0u, sbits = 0x80000000u;
                bool sig = false;
                const unsigned own = MALL ? 255u : 1u << xcd;
                u64 v = __hip_atomic_load(gp, __ATOMIC_RELAXED, __HIP_MEMORY_SCOPE_SYSTEM);
                for (unsigned it = 1;; ++it) {
                    const bool ok = (unsigned)(v >> 32) >= want;
                    const unsigned gm = ok ? (unsigned)v | (pass ^ 1u) : 0u;
                    const bool all = __all(ok), newer = __any(gm > best);
                    if (all && !newer) break;
                    if (MPOS == 3 && !sig && (((unsigned)(__ballot(ok) >> (rank * 8u)) & own) == own)) { sig = true; CTL[pass ? 20u : 19u] = 1u; }
                    if (GSLEEP && __builtin_popcountll(__ballot(ok)) < GTHR) __builtin_amdgcn_s_sleep(GSLEEP);
                    v = __hip_atomic_load(gp, __ATOMIC_RELAXED, __HIP_MEMORY_SCOPE_SYSTEM);
                    if (newer) {
                        best = wave_umax(gm);
                        sbits = best == BAD ? BAD : __float_as_uint(127.0f / __uint_as_float(best)) | (best >= 0x7f800000u ? 0x80000000u : 0u);
                    }
                    if (all) break;
                    if ((it & 63u) == 0u) {
                        if (rt() - t0 > DEADLINE_TICKS ||
                            __builtin_amdgcn_readfirstlane(__hip_atomic_load(tmo, __ATOMIC_RELAXED, __HIP_MEMORY_SCOPE_AGENT)) == call) {
                            __hip_atomic_store(tmo, call, __ATOMIC_RELAXED, __HIP_MEMORY_SCOPE_AGENT);
                            sbits = BAD;
                            break;
                        }
                    }
                }
                CTL[pass ? 16u : 19u] = sbits;
            }
            WAIT_LGKM0();
        }
        return;
    }

    const int wm = wave >> 1, wn = wave & 1;
    const int g = lane >> 4, i = lane & 15;
    const unsigned rb = rbp0 * 2u + ((unsigned)wm >> 1);
    const unsigned sidx = rb < 20u ? rb / 5u : 4u + (rb - 20u) / 3u;
    const unsigned sblk = rb < 20u ? rb % 5u : 5u + (rb - 20u) % 3u;
    const unsigned rloc = ((unsigned)wm & 1u) * 32u;
    const unsigned grow = moved ? rank * 512u + (5u + (mem & 1u) + ((unsigned)wm >> 1)) * 64u + rloc
                                : ((sidx + grp * 4u) & 7u) * 512u + sblk * 64u + rloc;
    const unsigned prow = ((sidx + grp * 4u + 4u) & 7u) * 512u + sblk * 64u + rloc;
    const bool remote = cb >= 4u;
    const bool watch = cb0 < 4u && !(OWNROWS && rbp0 == ((0xfdcau >> (mem * 4u)) & 15u));
    const unsigned ocol = cb * 128u + (unsigned)wn * 64u;
    const unsigned msg = (rbp0 * 4u + (remote ? mem : cb)) * 8u + (unsigned)wave;
    const __amdgpu_buffer_rsrc_t rm = mkrsrc(p.wsr[remote ? (cb - mem) & 7u : 0u], WS_USED);
    const __amdgpu_buffer_rsrc_t ro = mkrsrc(p.out, OUT_BYTES);
    GAS u64* g1 = (GAS u64*)(wsme + G1_OFF + xcd * 1024u + (sl * 8u + (unsigned)wave) * 8u);
    asm volatile("" : "+s"(g1));
    if (GWARM && lane == 0) __hip_atomic_store(g1 + (G0_OFF - G1_OFF) / 8, (u64)(call - 1u) << 32, __ATOMIC_RELAXED, __HIP_MEMORY_SCOPE_WORKGROUP);
    if (CPRIO) __builtin_amdgcn_s_setprio(CPRIO);
    if (watch) {
        const v4u mark = {0x80808080u, 0x80808080u, 0x80808080u, 0x80808080u};
        st128(mark, rm, lane * 16u, D_OFF + msg * 2048u);
        st128(mark, rm, lane * 16u + 1024u, D_OFF + msg * 2048u);
    }
    f32x4 acc[2][4];
#pragma unroll
    for (int m = 0; m < 2; ++m)
#pragma unroll
        for (int f = 0; f < 4; ++f) acc[m][f] = (f32x4){0.f, 0.f, 0.f, 0.f};
    const unsigned a_off0 = (unsigned)(32 * wm + i) * 128u + (unsigned)((g ^ (i & 7)) * 16);
    const unsigned a_off1 = (unsigned)(32 * wm + i) * 128u + (unsigned)(((4 + g) ^ (i & 7)) * 16);
    const unsigned b_off = 16384u + (unsigned)(i & 1) * 4096u + (unsigned)(g * 16 + ((8 * wn + (i >> 1)) ^ (8 * (i & 1)))) * 16u;
    bool dead = false;
    unsigned mycnt = 0;
    unsigned last = 3;
    const volatile LAS v2u* fullp = (const volatile LAS v2u*)(L8 + CTL_OFF + (lane < 4 ? lane : 4) * 8);
    WAIT_VM0();
    {
        v4u a1[2], b1[4];
        a1[0] = a1[1] = b1[0] = b1[1] = b1[2] = b1[3] = (v4u){0u, 0u, 0u, 0u};
        v2u f = {0u, 0u};
        const unsigned fa = ctl + (unsigned)(lane < 4 ? lane : 4) * 8u;
#define MM(A, B, mm, f0, f1) for (int f_ = f0; f_ < f1; ++f_) acc[mm][f_] = __builtin_amdgcn_mfma_f32_16x16x32_bf16(__builtin_bit_cast(bf16x8, A), __builtin_bit_cast(bf16x8, B[f_]), acc[mm][f_], 0, 0, 0)
#define RD6(A, B, pa, pb) asm volatile("ds_read_b128 %0, %6\n\tds_read_b128 %2, %7\n\tds_read_b128 %3, %7 offset:1024\n\tds_read_b128 %4, %7 offset:2048\n\t" \
                          "ds_read_b128 %5, %7 offset:3072\n\tds_read_b128 %1, %6 offset:2048" \
                          : "=&v"(A[0]), "=&v"(A[1]), "=&v"(B[0]), "=&v"(B[1]), "=&v"(B[2]), "=&v"(B[3]) : "v"(pa), "v"(pb) : "memory")
#pragma nounroll
        for (unsigned n = 0;; ++n) {
            bool av = min(f.x, f.y) > mycnt;
            unsigned m = (unsigned)__ballot(av) & 31u;
            const bool got = m != 0u;
            if (!got) {
                _Pragma("unroll") MM(a1[0], b1, 0, 0, 4);
                _Pragma("unroll") MM(a1[1], b1, 1, 0, 4);
                if (n == (unsigned)KSTEPS) break;
                while (true) {
                    f = *fullp;
                    av = min(f.x, f.y) > mycnt;
                    m = (unsigned)__ballot(av) & 31u;
                    if (m) break;
                    __builtin_amdgcn_s_sleep(CSLEEP);
                }
            }
            if (m & 16u) { dead = true; break; }
            unsigned key = av ? mycnt * 4u + (unsigned)lane : 1023u;
            key = min(key, (unsigned)__builtin_amdgcn_update_dpp(0, (int)key, 0xb1, 0xf, 0xf, true));
            key = min(key, (unsigned)__builtin_amdgcn_update_dpp(0, (int)key, 0x4e, 0xf, 0xf, true));
            const unsigned s = (unsigned)__builtin_amdgcn_readfirstlane(key) & 3u;
            const unsigned so = s * SLOT_BYTES;
            v4u a0[2], b0[4];
            RD6(a0, b0, so + a_off0, so + b_off);
            __builtin_amdgcn_sched_barrier(0);
            mycnt += ((unsigned)lane == s) ? 1u : 0u;
            const unsigned pa1 = so + a_off1, pb1 = so + b_off + 8192u, fr = ctl + 48u + s * 4u;
            if (got) {
                _Pragma("unroll") MM(a1[0], b1, 0, 0, 4);
                _Pragma("unroll") MM(a1[1], b1, 1, 0, 4);
            }
            __builtin_amdgcn_sched_barrier(0);
            RD6(a1, b1, pa1, pb1);
            asm volatile("s_waitcnt lgkmcnt(9)" : "+v"(a0[0]), "+v"(b0[0]), "+v"(b0[1]) :: "memory");
            _Pragma("unroll") MM(a0[0], b0, 0, 0, 2);
            __builtin_amdgcn_sched_barrier(0);
            asm volatile("s_waitcnt lgkmcnt(7)" : "+v"(b0[2]), "+v"(b0[3]) :: "memory");
            _Pragma("unroll") MM(a0[0], b0, 0, 2, 4);
            __builtin_amdgcn_sched_barrier(0);
            asm volatile("s_waitcnt lgkmcnt(6)\n\tds_read_b64 %1, %2" : "+v"(a0[1]), "=&v"(f) : "v"(fa) : "memory");
            _Pragma("unroll") MM(a0[1], b0, 1, 0, 4);
            __builtin_amdgcn_sched_barrier(0);
            asm volatile("s_waitcnt lgkmcnt(0)\n\t"
                         "s_mov_b64 exec, 1\n\t"
                         "ds_add_u32 %7, %8\n\t"
                         "s_mov_b64 exec, -1"
                         : "+v"(a1[0]), "+v"(a1[1]), "+v"(b1[0]), "+v"(b1[1]), "+v"(b1[2]), "+v"(b1[3]), "+v"(f)
                         : "v"(fr), "v"(1u) : "memory");
            __builtin_amdgcn_sched_barrier(0);
        }
    }
    TS(ts5);

    {
        float mx = 0.f;
#pragma unroll
        for (int m = 0; m < 2; ++m)
#pragma unroll
            for (int f = 0; f < 4; ++f)
#pragma unroll
                for (int r = 0; r < 4; ++r) mx = fmaxf(mx, acc[m][f][r]);
        const unsigned wmx = dead ? BAD : wave_umax(__float_as_uint(mx));
        if (lane == 0) {
            if (FASTG) __hip_atomic_store(g1 + (G0_OFF - G1_OFF) / 8, ((u64)call << 32) | wmx, __ATOMIC_RELAXED, __HIP_MEMORY_SCOPE_WORKGROUP);
            __hip_atomic_store(g1, ((u64)call << 32) | wmx, __ATOMIC_RELAXED, __HIP_MEMORY_SCOPE_SYSTEM);
        }
    }
    TS(ts6);
    if (CPRIO) __builtin_amdgcn_s_setprio(0);
    asm volatile("" : "+v"(acc[0][0]), "+v"(acc[0][1]), "+v"(acc[0][2]), "+v"(acc[0][3]),
                      "+v"(acc[1][0]), "+v"(acc[1][1]), "+v"(acc[1][2]), "+v"(acc[1][3]) :: "memory");
    const unsigned xb = ((unsigned)wave & 3u) * SLOT_BYTES + ((unsigned)wave >> 2) * 16384u;
    const unsigned ovo = (unsigned)g * 2048u + (unsigned)i * 4u;
    if (remote) {
        unsigned mk = 0;
#pragma unroll
        for (int j = 7; j >= 0; --j)
#pragma unroll
            for (int f = 3; f >= 0; --f) mk = __builtin_amdgcn_alignbit(mk, __float_as_uint(acc[j >> 2][f][j & 3]), 31u);
        mk = ~mk;
        if (mk == 0u) { mk = 1u; acc[0][0][0] = 0.f; }
        if (MPOS == 0) __builtin_amdgcn_raw_buffer_store_b32(mk, rm, lane * 4u, M_OFF + msg * 256u, 17);
        unsigned o[8];
        const unsigned cnt = opaque(positions(mk, o));
        for (unsigned it = 1; !dead && CTL[12 + (wave & 3)] < 128u; ++it)
            if ((it & 63u) == 0u && CTL[8] != 0u) dead = true;
        asm volatile("" ::: "memory");
        LAS f32x4* const xp = (LAS f32x4*)(L8 + xb + 16 + lane * 16);
        for (unsigned q = cnt + (unsigned)lane; q < max(1024u, cnt + 16u); q += 64u) *(LAS float*)(L8 + xb + 16 + q * 4u) = 0.f;
#pragma unroll
        for (int j = 0; j < 8; ++j) {
            unsigned a = xb + 16u + o[j] * 4u;
#pragma unroll
            for (int f = 0; f < 4; ++f) {
                const float v = acc[j >> 2][f][j & 3];
                const unsigned sg = (unsigned)(__float_as_int(v) >> 31);
                asm volatile("ds_write_b32 %0, %1 offset:%2" :: "v"(a | (sg & 0x40000u)), "v"(v), "n"(f * 4) : "memory");
                a += sg << 2;
            }
        }
        if (MPOS == 1) __builtin_amdgcn_raw_buffer_store_b32(mk, rm, lane * 4u, M_OFF + msg * 256u, 17);
        f32x4 d[8];
#pragma unroll
        for (int c = 0; c < 8; ++c) d[c] = xp[c * 64];
        asm volatile("s_waitcnt lgkmcnt(0)" : "+v"(d[0]), "+v"(d[1]), "+v"(d[2]), "+v"(d[3]),
                                                       "+v"(d[4]), "+v"(d[5]), "+v"(d[6]), "+v"(d[7]) :: "memory");
        const unsigned lim = (unsigned)(lane & ~3) * 4u;
        const unsigned dso = opaque(D_OFF + msg * 2048u);
        unsigned sb;
        if (MPOS == 3) {
            while (__builtin_amdgcn_readfirstlane(CTL[20] | CTL[16]) == 0u) { }
            __builtin_amdgcn_raw_buffer_store_b32(mk, rm, lane * 4u, M_OFF + msg * 256u, 17);
        }
        do { sb = __builtin_amdgcn_readfirstlane(CTL[16]); } while (sb == 0u);
        sb = opaque(sb);
        TS(ts7);
        if (sb != BAD) {
            if (MPOS == 2) __builtin_amdgcn_raw_buffer_store_b32(mk, rm, lane * 4u, M_OFF + msg * 256u, 17);
            const float inv = __uint_as_float(sb);
            v4u q;
#pragma unroll
            for (int c = 0; c < 4; ++c) q[c] = quant4(d[c], inv, 12582912.f);
            asm volatile("" : "+v"(q) :: "memory");
            int acked = 0;
            while (acked < wave + 1 - OUT_WIN) acked = (int)__builtin_amdgcn_readfirstlane(CTL[17]);
            st128(q, rm, lane * 16u, dso);
#pragma unroll
            for (int c = 4; c < 8; ++c) {
                if (cnt <= 256u * c) break;
                const unsigned e = quant4(d[c], inv, 12582912.f);
                if (lim + 256u * c < cnt) __builtin_amdgcn_raw_buffer_store_b32(e, rm, lane * 4u + 256u * c, dso, 17);
            }
        }
        asm volatile("s_waitcnt vmcnt(0)\n\t"
                     "s_mov_b64 exec, 1\n\t"
                     "ds_add_u32 %0, %1 offset:68\n\t"
                     "s_mov_b64 exec, -1" :: "v"(ctl), "v"(1u) : "memory");
        return;
    }

#pragma unroll
    for (int m = 0; m < 2; ++m)
#pragma unroll
        for (int f = 0; f < 4; ++f)
#pragma unroll
            for (int r = 0; r < 4; ++r) acc[m][f][r] = __int_as_float(max(__float_as_int(acc[m][f][r]), 0));
    asm volatile("" : "+v"(acc[0][0]), "+v"(acc[0][1]), "+v"(acc[0][2]), "+v"(acc[0][3]),
                      "+v"(acc[1][0]), "+v"(acc[1][1]), "+v"(acc[1][2]), "+v"(acc[1][3]) :: "memory");
    unsigned sb;
    do { sb = __builtin_amdgcn_readfirstlane(CTL[16]); } while (sb == 0u);
    TS(ts7);
    dead = sb == BAD;
    unsigned mk = __builtin_amdgcn_raw_buffer_load_b32(rm, lane * 4u, M_OFF + msg * 256u, 17);
    {
        const float inv = dead ? 0.f : __uint_as_float(sb);
        const float magic = dead ? 12583040.f : 12582912.f;
#pragma unroll
        for (int m = 0; m < 2; ++m) {
            const f32x2 iv = {inv, inv}, mg = {magic, magic};
            f32x2 z[4][2];
#pragma unroll
            for (int f = 0; f < 4; ++f) {
                z[f][0] = __builtin_elementwise_fma((f32x2){acc[m][f][0], acc[m][f][1]}, iv, mg);
                z[f][1] = __builtin_elementwise_fma((f32x2){acc[m][f][2], acc[m][f][3]}, iv, mg);
            }
#pragma unroll
            for (int r = 0; r < 4; ++r) {
                const unsigned lo = __builtin_amdgcn_perm(__float_as_uint(z[1][r >> 1][r & 1]), __float_as_uint(z[0][r >> 1][r & 1]), 0x0c0c0400u);
                const unsigned hi = __builtin_amdgcn_perm(__float_as_uint(z[3][r >> 1][r & 1]), __float_as_uint(z[2][r >> 1][r & 1]), 0x04000c0cu);
                __builtin_amdgcn_raw_buffer_store_b32(lo | hi, ro, ovo + r * 512u, (grow + 16u * m) * 512u + ocol, OAUX);
            }
        }
    }
    if (watch && !dead) {
        unsigned ra[8], sel[8], o[8];
        bool late = false;
        for (unsigned it = 1; __any(mk == 0u); ++it) {
            mk = __builtin_amdgcn_raw_buffer_load_b32(rm, lane * 4u, M_OFF + msg * 256u, 17);
            if ((it & 63u) == 0u && (rt() - t0 > DEADLINE_TICKS ||
                __builtin_amdgcn_readfirstlane(__hip_atomic_load(tmo, __ATOMIC_RELAXED, __HIP_MEMORY_SCOPE_AGENT)) == call)) { late = true; break; }
        }
        __builtin_amdgcn_raw_buffer_store_b32(0u, rm, lane * 4u, M_OFF + msg * 256u, 17);
        const unsigned cnt = positions(mk, o);
#pragma unroll
        for (int j = 0; j < 8; ++j) {
            const unsigned ps = (((mk >> (4 * j)) & 15u) * 0x00204081u) & 0x01010101u;
            unsigned rk = (ps << 8) | (o[j] & 3u);
            rk += rk << 8;
            asm("" : "+v"(rk));
            rk += rk << 16;
            const unsigned am = ps + 0x0f0f0f0fu;
            sel[j] = (rk & ~am) | (0x0c0c0c0cu & am);
            ra[j] = xb + (o[j] & ~3u);
        }
        unsigned vo1 = (unsigned)lane * 16u + 1024u < cnt ? (unsigned)lane * 16u + 1024u : 0x80000000u;
        asm volatile("" : "+v"(ra[0]), "+v"(ra[1]), "+v"(ra[2]), "+v"(ra[3]), "+v"(ra[4]), "+v"(ra[5]), "+v"(ra[6]), "+v"(ra[7]),
                          "+v"(sel[0]), "+v"(sel[1]), "+v"(sel[2]), "+v"(sel[3]), "+v"(sel[4]), "+v"(sel[5]), "+v"(sel[6]), "+v"(sel[7]), "+v"(vo1) :: "memory");
        const unsigned pso = opaque(prow * 512u + ocol);
        while (!late) {
            unsigned got, t, t2;
#define DLD(a, b) "buffer_load_dwordx4 v[" #a "], %3, %5, %6 offen sc0 sc1\n\tbuffer_load_dwordx4 v[" #b "], %4, %5, %6 offen sc0 sc1\n\ts_sleep %9\n\t"
#define DCK(k, r0, r1, r2, r3, r4, r5, r6, r7) "s_waitcnt vmcnt(%10)\n\t" \
            "v_or3_b32 %1, v" #r0 ", v" #r1 ", v" #r2 "\n\tv_or3_b32 %2, v" #r3 ", v" #r4 ", v" #r5 "\n\tv_or3_b32 %1, %1, v" #r6 ", v" #r7 "\n\t" \
            "v_or_b32 %1, %1, %2\n\tv_and_b32 %1, 0x80808080, %1\n\tv_cmp_ne_u32 vcc, 0, %1\n\ts_cbranch_vccz .Ldg" #k "%=\n\t" DLD(r0:r3, r4:r7)
#define DWR(k, r0, r1, r2, r3, r4, r7) ".Ldg" #k "%=:\n\tds_write2st64_b32 %7, v" #r0 ", v" #r1 " offset1:1\n\tds_write2st64_b32 %7, v" #r2 ", v" #r3 " offset0:2 offset1:3\n\t" \
            "ds_write_b128 %8, v[" #r4 ":" #r7 "] offset:1024\n\ts_mov_b32 %0, 1\n\t"
            asm volatile(DLD(104:107, 108:111)
                         "s_movk_i32 %0, 100\n"
                         ".Ldl%=:\n\t"
                         DCK(0, 104, 105, 106, 107, 108, 109, 110, 111)
                         "s_sub_u32 %0, %0, 1\n\t"
                         "s_cbranch_scc0 .Ldl%=\n\t"
                         "s_mov_b32 %0, 0\n\t"
                         "s_branch .Lde%=\n"
                         DWR(0, 104, 105, 106, 107, 108, 111)
                         "\n.Lde%=:"
                         : "=&s"(got), "=&v"(t), "=&v"(t2)
                         : "v"(lane * 16u), "v"(vo1), "s"(rm), "s"(D_OFF + msg * 2048u), "v"(xb + lane * 4u), "v"(xb + lane * 16u), "n"(DPD > 1 ? DPS : 0), "n"(2 * DPD - 2)
                         : "vcc", "scc", "memory", "v104", "v105", "v106", "v107", "v108", "v109", "v110", "v111", "v112", "v113", "v114", "v115",
                           "v116", "v117", "v118", "v119", "v120", "v121", "v122", "v123", "v124", "v125", "v126", "v127");
            if (got) break;
            if (rt() - t0 > DEADLINE_TICKS ||
                __builtin_amdgcn_readfirstlane(__hip_atomic_load(tmo, __ATOMIC_RELAXED, __HIP_MEMORY_SCOPE_AGENT)) == call) late = true;
        }
        if (late) {
            __hip_atomic_store(tmo, call, __ATOMIC_RELAXED, __HIP_MEMORY_SCOPE_AGENT);
            __builtin_amdgcn_raw_buffer_store_b32(0x80808080u, ro, ovo, prow * 512u + ocol, 17);
        } else {
            v2u e[8];
#pragma unroll
            for (int j = 0; j < 8; ++j) {
                const LAS unsigned* rp = (const LAS unsigned*)(L8 + ra[j]);
                e[j] = (v2u){rp[0], rp[1]};
            }
            __builtin_amdgcn_sched_barrier(0);
#pragma unroll
            for (int j = 0; j < 8; ++j)
                __builtin_amdgcn_raw_buffer_store_b32(__builtin_amdgcn_perm(e[j].y, e[j].x, sel[j]), ro, ovo + (j & 3) * 512u, pso + (j >> 2) * 8192u, OAUX);
        }
    }
}

extern "C" void kernel_launch(int rank, int nranks, ncclComm_t comm, hipStream_t stream,
                              void* const* d_in, const int* in_sizes, int n_in,
                              void* d_out, int out_size, const int* in_dtypes, int out_dtype,
                              void* const* d_out_all, void* const* d_ws_all, size_t ws_bytes) {
    if (nranks != NR || n_in < 2 || ws_bytes < (size_t)WS_USED + 4096u || (unsigned)rank >= (unsigned)NR ||
        in_sizes[0] != 512 * 4096 || in_sizes[1] != 4096 * 512 || out_size != 4096 * 512) {
        fprintf(stderr, "unexpected problem configuration\n");
        return;
    }
    Params p;
    p.x = d_in[0];
    p.w = d_in[1];
    for (int d = 0; d < NR; ++d) p.wsr[d] = (char*)d_ws_all[(rank + d) & 7];
    p.out = d_out;
    p.rank = rank;
    ag_gemm_kernel<<<dim3(NWG), dim3(1024), 0, stream>>>(p);
}
```

```cpp
#include <hip/hip_runtime.h>
#include <rccl/rccl.h>
#include <cstdio>
#include <cstdint>

#define LAS __attribute__((address_space(3)))
#define GAS __attribute__((address_space(1)))
typedef unsigned v4u __attribute__((ext_vector_type(4)));
typedef unsigned v2u __attribute__((ext_vector_type(2)));
typedef float f32x4 __attribute__((ext_vector_type(4)));
typedef float f32x2 __attribute__((ext_vector_type(2)));
typedef __bf16 bf16x8 __attribute__((ext_vector_type(8)));
typedef unsigned long long u64;


constexpr int NR = 8;
constexpr int CU_PER_DEST = 8, WV_PER_CU = 3;
constexpr int SELF_CU = 16, SELF_WV = 4;
constexpr int WIN = 6;
constexpr int AP = 4;
constexpr int OUT_WIN = 4;
constexpr unsigned TN = 2;
constexpr unsigned SI = 4;
constexpr int NWG = 256;
constexpr int NGW = 128;
constexpr int KSTEPS = 64;
constexpr int NSLOT = 4;
constexpr unsigned NRB = 36;
constexpr unsigned A_OFF = 0;
constexpr unsigned B_OFF = 18u << 20;
constexpr unsigned F_OFF = 26u << 20;
constexpr unsigned FA_OFF = F_OFF;
constexpr unsigned FB_OFF = F_OFF + 12288;
constexpr unsigned Q_OFF = F_OFF + 16384;
constexpr unsigned TM_OFF = F_OFF + 18432;
constexpr unsigned R_OFF = F_OFF + 20480;
constexpr unsigned G1_OFF = F_OFF + 24576;
constexpr unsigned CT_OFF = F_OFF + 32768;
constexpr unsigned G0_OFF = F_OFF + 49152;
constexpr unsigned G2_OFF = F_OFF + 65536;
constexpr unsigned M_OFF = F_OFF + (512u << 10);
constexpr unsigned D_OFF = 27u << 20;
constexpr unsigned WS_USED = 28u << 20;
constexpr unsigned X_BYTES = 512u * 4096u * 2u;
constexpr unsigned OUT_BYTES = 4096u * 512u;
constexpr int SLOT_BYTES = 32768;
constexpr int CTL_OFF = NSLOT * SLOT_BYTES;
constexpr int LDS_BYTES = CTL_OFF + 128;
constexpr u64 DEADLINE_TICKS = 40000000ull;
constexpr int RSRC3 = 0x00020000;
constexpr unsigned BAD = 0xffffffffu;
static_assert(LDS_BYTES <= 160 * 1024, "LDS");

struct Params {
    const void* x;
    const void* w;
    void* out;
    long long rank;
    char* wsr[NR];
};

__device__ __forceinline__ unsigned opaque(unsigned v) { asm volatile("" : "+s"(v)); return v; }
__device__ __forceinline__ u64 rt() { return __builtin_amdgcn_s_memrealtime(); }
__device__ __forceinline__ __amdgpu_buffer_rsrc_t mkrsrc(const void* p, unsigned bytes) {
    return __builtin_amdgcn_make_buffer_rsrc((void*)p, (short)0, (int)bytes, RSRC3);
}
#define WAIT_VM0() asm volatile("s_waitcnt vmcnt(0)" ::: "memory")
#define WAIT_VM0_B() do { asm volatile("" ::: "memory"); __builtin_amdgcn_s_waitcnt(0x0F70); asm volatile("" ::: "memory"); } while (0)
#define WAIT_LGKM0() asm volatile("s_waitcnt lgkmcnt(0)" ::: "memory")

__device__ __forceinline__ void st128(const v4u d, const __amdgpu_buffer_rsrc_t r, unsigned vo, unsigned so) {
    __builtin_amdgcn_raw_buffer_store_b128(d, r, vo, so, 17);
    asm volatile("s_nop 1" :: "v"(d) : "memory");
}


__device__ __forceinline__ unsigned wave_umax(unsigned v) {
    v = max(v, (unsigned)__builtin_amdgcn_update_dpp(0, (int)v, 0x111, 0xf, 0xf, true));
    v = max(v, (unsigned)__builtin_amdgcn_update_dpp(0, (int)v, 0x112, 0xf, 0xf, true));
    v = max(v, (unsigned)__builtin_amdgcn_update_dpp(0, (int)v, 0x114, 0xf, 0xf, true));
    v = max(v, (unsigned)__builtin_amdgcn_update_dpp(0, (int)v, 0x118, 0xf, 0xf, true));
    const unsigned a = __builtin_amdgcn_readlane(v, 15), b = __builtin_amdgcn_readlane(v, 31);
    const unsigned c = __builtin_amdgcn_readlane(v, 47), d = __builtin_amdgcn_readlane(v, 63);
    return max(max(a, b), max(c, d));
}

__device__ __forceinline__ unsigned wave_scan(unsigned v) {
    v += (unsigned)__builtin_amdgcn_update_dpp(0, (int)v, 0x111, 0xf, 0xf, true);
    v += (unsigned)__builtin_amdgcn_update_dpp(0, (int)v, 0x112, 0xf, 0xf, true);
    v += (unsigned)__builtin_amdgcn_update_dpp(0, (int)v, 0x114, 0xf, 0xf, true);
    v += (unsigned)__builtin_amdgcn_update_dpp(0, (int)v, 0x118, 0xf, 0xf, true);
    v += (unsigned)__builtin_amdgcn_update_dpp(0, (int)v, 0x142, 0xa, 0xf, false);
    v += (unsigned)__builtin_amdgcn_update_dpp(0, (int)v, 0x143, 0xc, 0xf, false);
    return v;
}

__device__ __forceinline__ unsigned positions(unsigned mk, unsigned (&o)[8]) {
    unsigned x = mk;
    x -= (x >> 1) & 0x55555555u;
    x = (x & 0x33333333u) + ((x >> 2) & 0x33333333u);
    unsigned base = 0;
    unsigned c[2], e[2], cl[2], el[2];
    c[0] = x & 0x0f0f0f0fu;
    c[1] = (x >> 4) & 0x0f0f0f0fu;
#pragma unroll
    for (int h = 0; h < 2; ++h) {
        e[h] = wave_scan(c[h]) - c[h];
        cl[h] = __builtin_amdgcn_readlane(c[h], 63);
        el[h] = __builtin_amdgcn_readlane(e[h], 63);
    }
#pragma unroll
    for (int j = 0; j < 8; ++j) {
        o[j] = base + ((e[j & 1] >> (8 * (j >> 1))) & 255u);
        base += ((cl[j & 1] >> (8 * (j >> 1))) & 255u) + ((el[j & 1] >> (8 * (j >> 1))) & 255u);
    }
    return base;
}

__device__ __forceinline__ unsigned quant4(const f32x4 d, float inv, float magic) {
    const f32x2 iv = {inv, inv}, mg = {magic, magic};
    const f32x2 a = __builtin_elementwise_fma((f32x2){d[0], d[1]}, iv, mg), b = __builtin_elementwise_fma((f32x2){d[2], d[3]}, iv, mg);
    return __builtin_amdgcn_perm(__float_as_uint(a[1]), __float_as_uint(a[0]), 0x0c0c0400u) |
           __builtin_amdgcn_perm(__float_as_uint(b[1]), __float_as_uint(b[0]), 0x04000c0cu);
}

__global__ void __launch_bounds__(1024, 1) ag_gemm_kernel(Params p) {
    __shared__ __attribute__((aligned(16))) unsigned char lds[LDS_BYTES];
    LAS unsigned char* L8 = (LAS unsigned char*)lds;
    volatile LAS unsigned* CTL = (volatile LAS unsigned*)(L8 + CTL_OFF);
    const int tid = threadIdx.x;
    const int wave = __builtin_amdgcn_readfirstlane(tid >> 6);
    const int lane = tid & 63;
    asm volatile("" :: "s"(p.x), "s"(p.w), "s"(p.rank), "s"(p.wsr[0]), "s"(p.wsr[7]));
    const unsigned rank = (unsigned)p.rank & 7u;
    const unsigned grp = rank >> 2, mem = rank & 3u;
    const unsigned b = blockIdx.x;
    char* const wsme = p.wsr[0];
    GAS unsigned* ctr = (GAS unsigned*)(wsme + CT_OFF + (b * 16 + wave) * 4);
    GAS unsigned* tmo = (GAS unsigned*)(wsme + TM_OFF);
    const unsigned ctl = (unsigned)CTL_OFF;

    if (b < (unsigned)(NWG - NGW)) {
        if (b - 56u < 8u) {
            if (wave >= 1) return;
            const u64 t0 = rt();
            const unsigned j = b - 56u;
            const unsigned call = __builtin_amdgcn_readfirstlane(__hip_atomic_load(ctr, __ATOMIC_RELAXED, __HIP_MEMORY_SCOPE_AGENT)) + 1u;
            if (lane == 0) __hip_atomic_store(ctr, call, __ATOMIC_RELAXED, __HIP_MEMORY_SCOPE_AGENT);
            GAS u64* const dstp = (GAS u64*)(p.wsr[lane & 7] + G2_OFF + (unsigned)(lane >> 3) * 8192u + (rank * 8u + j) * 128u);
            const __amdgpu_buffer_rsrc_t rs = mkrsrc(wsme, WS_USED);
#pragma nounroll
            for (unsigned pass = opaque(0u); pass < 2u; pass = opaque(pass + 1u)) {
                const unsigned want = pass ? call : 0u;
                unsigned gm;
                for (unsigned it = 1;; ++it) {
                    const v4u v = __builtin_amdgcn_raw_buffer_load_b128(rs, lane * 16u, ((it & 255u) ? G0_OFF : G1_OFF) + j * 1024u, 17);
                    gm = max(v.x, v.z);
                    if (__all(v.y >= want && v.w >= want)) break;
                    if ((it & 255u) == 0u) {
                        if (rt() - t0 > DEADLINE_TICKS ||
                            __builtin_amdgcn_readfirstlane(__hip_atomic_load(tmo, __ATOMIC_RELAXED, __HIP_MEMORY_SCOPE_AGENT)) == call) {
                            __hip_atomic_store(tmo, call, __ATOMIC_RELAXED, __HIP_MEMORY_SCOPE_AGENT);
                            gm = BAD;
                            break;
                        }
                    }
                }
                gm = wave_umax(gm);
                __hip_atomic_store(dstp + (pass ? 0 : 8), ((u64)call << 32) | gm, __ATOMIC_RELAXED, __HIP_MEMORY_SCOPE_SYSTEM);
            }
            return;
        }
        if (b - 81u < 7u) {
            const unsigned d = b - 80u, src = (rank + d) & 7u;
            const unsigned call_v = __hip_atomic_load(ctr, __ATOMIC_RELAXED, __HIP_MEMORY_SCOPE_AGENT) + 1u;
            const __amdgpu_buffer_rsrc_t rs = mkrsrc(wsme, WS_USED);
            const bool same = (src >> 2) == grp;
            __builtin_amdgcn_s_sleep(127);
            const v4u z = {0u, 0u, 0u, 0u};
            for (unsigned q = (unsigned)wave; q < 160u * TN; q += 16u) {
                const unsigned ks = 64u - 4u * TN + q / 40u, r = q % 40u;
                const unsigned so = same ? A_OFF + (ks * NRB + (src & 3u) * 5u) * 8192u + r * 1024u
                                  : r < 24u ? A_OFF + (ks * NRB + 20u + (src & 3u) * 3u) * 8192u + r * 1024u
                                            : B_OFF + (ks * 8u + 4u + (src & 3u)) * 16384u + (r - 24u) * 1024u;
                st128(z, rs, lane * 16u, so);
            }
            WAIT_VM0_B();
            __builtin_amdgcn_s_barrier();
            if (tid == 0) {
                __hip_atomic_store(ctr, call_v, __ATOMIC_RELAXED, __HIP_MEMORY_SCOPE_AGENT);
                __hip_atomic_store((GAS unsigned*)(p.wsr[d] + R_OFF + (8u - d) * 128u), call_v, __ATOMIC_RELAXED, __HIP_MEMORY_SCOPE_SYSTEM);
            }
            return;
        }
        const bool self = b >= 64u;
        const unsigned dsel = self ? 0u : 1u + (b >> 3);
        const unsigned sub = self ? b - 64u : (b & 7u);
        if (self ? (sub >= (unsigned)SELF_CU || wave >= SELF_WV) : (dsel >= 8u || wave >= WV_PER_CU)) return;
        const unsigned ncu = self ? (unsigned)SELF_CU : (unsigned)CU_PER_DEST;
        const unsigned nwv = self ? (unsigned)(SELF_CU * SELF_WV) : (unsigned)(CU_PER_DEST * WV_PER_CU);
        const unsigned wv = (unsigned)wave * ncu + sub;
        const unsigned dst = (rank + dsel) & 7u;
        const bool same = (dst >> 2) == grp;
        const unsigned nper = self ? 16u : 5u;
        const unsigned kmul = self ? 4096u : 13108u;
        const unsigned nx = self ? 8u : same ? 5u : 3u;
        const unsigned xb0 = same ? 0u : 5u;
        const unsigned drb0 = same ? mem * 5u : 20u + mem * 3u;
        const unsigned nunits = nper * KSTEPS;
        char* const wsd = p.wsr[dsel];
        const __amdgpu_buffer_rsrc_t rd = mkrsrc(wsd, WS_USED);
        const __amdgpu_buffer_rsrc_t rx = mkrsrc(p.x, X_BYTES), rw = mkrsrc(p.w, X_BYTES);
        const unsigned xoff = (unsigned)(lane >> 3) * 8192u + (unsigned)((lane & 7) ^ (lane >> 3)) * 16u;
        const unsigned q = lane >> 4, i = lane & 15;
        const unsigned woff = q * 8192u + i * 16u;
        GAS unsigned* qhead = (GAS unsigned*)(wsme + Q_OFF + dsel * 128u);
        v4u cur[8];
        auto load_unit = [&](unsigned u) {
            const unsigned ks = (u * kmul) >> 16, idx = u - ks * nper;
            if (idx < nx) {
                const unsigned so = (xb0 + idx) * (64u * 8192u) + ks * 128u;
#pragma unroll
                for (int t = 0; t < 8; ++t) cur[t] = __builtin_amdgcn_raw_buffer_load_b128(rx, xoff, so + t * 65536u, 0);
            } else {
                const unsigned wi = idx - nx;
                const unsigned so = (ks * 64u + (wi & 1u) * 32u) * 1024u + (self ? (wi >> 1) : (dst & 3u)) * 256u;
#pragma unroll
                for (int t = 0; t < 8; ++t) cur[t] = __builtin_amdgcn_raw_buffer_load_b128(rw, woff + (t & 3) * 1024u, so + (t >> 2) * 4096u, 0);
            }
        };
        const int wbias = self ? (1 << 24) : WIN - 1;
        unsigned u = wv, call = 0u;
        const GAS unsigned* rdy = (const GAS unsigned*)(wsme + R_OFF + dsel * 128u);
#pragma nounroll
        while (true) {
            load_unit(u);
            const unsigned call_v = __hip_atomic_load(ctr, __ATOMIC_RELAXED, __HIP_MEMORY_SCOPE_AGENT) + 1u;
            const bool first = call == 0u;
            unsigned skip = 0u;
            if (first) {
                if (tid < 2) CTL[tid] = tid ? 0u : (self ? (unsigned)SELF_WV : (unsigned)WV_PER_CU);
                WAIT_LGKM0();
                __builtin_amdgcn_s_barrier();
            } else if (!self && u >= 300u - 20u * TN) {
                const u64 t0 = rt();
                while (__builtin_amdgcn_readfirstlane(__hip_atomic_load(rdy, __ATOMIC_RELAXED, __HIP_MEMORY_SCOPE_SYSTEM)) < call)
                    if (rt() - t0 > DEADLINE_TICKS) { skip = 0x80000000u; break; }
                if (skip) __hip_atomic_store(tmo, call, __ATOMIC_RELAXED, __HIP_MEMORY_SCOPE_AGENT);
            }
            const unsigned ks = (u * kmul) >> 16, idx = u - ks * nper;
            const bool isx = idx < nx;
            const unsigned drb = idx < 5u ? drb0 + idx : (mem & 1u) ? (idx == 5u ? 34u : 26u + idx) : 27u + idx;
            const unsigned wi = idx - nx;
            const unsigned dcb = self ? (wi >> 1) : 4u + mem, kb = wi & 1u;
            unsigned so = isx ? A_OFF + (ks * NRB + drb) * 8192u : B_OFF + (ks * 8u + dcb) * 16384u + kb * 8192u;
            unsigned so2 = so + 4096u;
            unsigned vo = (unsigned)lane * 16u | skip, vo2 = isx ? vo : vo ^ 128u;
            const unsigned fpos = (ks & 3u) * 128u + (ks >> 2) * 8u;
            unsigned foff = isx ? FA_OFF + (drb >> 1) * 512u + fpos + (drb & 1u) * 4u : FB_OFF + dcb * 512u + fpos + kb * 4u;
            asm volatile("" : "+s"(so), "+s"(so2), "+v"(vo), "+v"(vo2), "+v"(foff));
            WAIT_VM0_B();
            unsigned seq = (unsigned)wave;
            if (!first)
                asm volatile("s_mov_b64 exec, 1\n\t"
                             "ds_add_rtn_u32 %0, %1, %2\n\t"
                             "s_mov_b64 exec, -1"
                             : "+v"(seq) : "v"(ctl), "v"(1u) : "memory");
            v4u o[8];
            if (isx) {
#pragma unroll
                for (int t = 0; t < 8; ++t) o[t] = cur[t];
            } else {
#pragma unroll
                for (int d = 0; d < 4; ++d) {
#pragma unroll
                    for (int e = 0; e < 4; ++e) {
                        o[2 * d][e] = __builtin_amdgcn_perm(cur[2 * e + 1][d], cur[2 * e][d], 0x05040100u);
                        o[2 * d + 1][e] = __builtin_amdgcn_perm(cur[2 * e + 1][d], cur[2 * e][d], 0x07060302u);
                    }
                }
            }
            asm volatile("s_waitcnt lgkmcnt(0)" : "+v"(seq) :: "memory");
            const int need = 8 * (int)__builtin_amdgcn_readfirstlane(seq) - wbias;
            int acked = 0;
            unsigned tk;
#pragma unroll
            for (int t = 0; t < 8; ++t) {
                if (t < WIN) {
                    while (acked < need + t) acked = (int)__builtin_amdgcn_readfirstlane(CTL[1]);
                } else {
                    asm volatile("s_waitcnt vmcnt(%2)\n\t"
                                 "s_mov_b64 exec, 1\n\t"
                                 "ds_add_u32 %0, %1 offset:4\n\t"
                                 "s_mov_b64 exec, -1" :: "v"(ctl), "v"(1u), "n"(WIN - 1 + (AP > t - WIN && AP < t ? 1 : 0)) : "memory");
                }
                if (t == AP)
                    asm volatile("s_mov_b64 exec, 1\n\t"
                                 "global_atomic_add %0, %1, %2, %3 sc0\n\t"
                                 "s_mov_b64 exec, -1"
                                 : "=&v"(tk) : "v"(0u), "v"(1u), "s"(qhead) : "memory");
                st128(o[t], rd, (t < 4 ? vo : vo2) + (t & 3) * 1024u, t < 4 ? so : so2);
            }
            call = __builtin_amdgcn_readfirstlane(call_v);
#define ACK(j) ".if %6 >= 8 - " #j "\n\t.if %7 > " #j "\n\ts_waitcnt vmcnt(8 - " #j ")\n\t.else\n\ts_waitcnt vmcnt(7 - " #j ")\n\t.endif\n\tds_add_u32 %5, %1 offset:4\n\t.endif\n\t"
            asm volatile("s_mov_b64 exec, 1\n\t"
                         ACK(0) ACK(1) ACK(2) ACK(3) ACK(4) ACK(5) ACK(6)
                         "s_waitcnt vmcnt(0)\n\t"
                         "global_store_dword %2, %3, %4 sc0 sc1\n\t"
                         "ds_add_u32 %5, %1 offset:4\n\t"
                         "s_mov_b64 exec, -1"
                         : "+v"(tk) : "v"(1u), "v"(foff), "v"(skip ? call - 1u : call), "s"(wsd), "v"(ctl), "n"(WIN), "n"(AP) : "memory");
            u = __builtin_amdgcn_readfirstlane(tk) - ((call - 1u) * nunits - nwv);
            if (u >= nunits) break;
        }
        if (lane == 0) __hip_atomic_store(ctr, call, __ATOMIC_RELAXED, __HIP_MEMORY_SCOPE_AGENT);
        return;
    }

    const u64 t0 = rt();
    const unsigned call_v = __hip_atomic_load(ctr, __ATOMIC_RELAXED, __HIP_MEMORY_SCOPE_AGENT) + 1u;
    if (tid < 32) CTL[tid] = tid == 18 ? BAD : 0u;
    const unsigned gb = b - (unsigned)(NWG - NGW);
    const unsigned xcd = gb & 7u, sl = gb >> 3;
    const unsigned rbp0 = (xcd >> 1) * 4u + (sl & 3u);
    const unsigned cbi = sl >> 2;
    const unsigned cb0 = (cbi & 1u) + 2u * (xcd & 1u) + 4u * (cbi >> 1);
    const bool moved = cb0 >= 4u && rbp0 == ((0xfdcau >> ((cb0 - 4u) * 4u)) & 15u);
    const unsigned rbp = moved ? 16u : rbp0;
    const unsigned cb = moved ? cb0 - 4u : cb0;
    WAIT_LGKM0();
    __builtin_amdgcn_s_barrier();
    const unsigned call = __builtin_amdgcn_readfirstlane(call_v);
    if (lane == 0) __hip_atomic_store(ctr, call, __ATOMIC_RELAXED, __HIP_MEMORY_SCOPE_AGENT);

    if (wave >= 8) {
        __builtin_amdgcn_s_setprio(2);
        const unsigned s = (unsigned)wave & 3u, h = ((unsigned)wave >> 2) & 1u;
        const __amdgpu_buffer_rsrc_t rs = mkrsrc(wsme, WS_USED);
        const GAS u64* fp = (const GAS u64*)(wsme + (h ? FB_OFF + cb * 512u : FA_OFF + rbp * 512u) + s * 128u + (lane & 15) * 8u);
        const unsigned src0 = h ? B_OFF + (s * 8u + cb) * 16384u : A_OFF + (s * NRB + rbp * 2u) * 8192u;
        const unsigned sstep = h ? 4u * 8u * 16384u : 4u * NRB * 8192u;
        LAS unsigned char* dst = L8 + s * SLOT_BYTES + h * 16384u;
        const GAS u64* gp = (const GAS u64*)(wsme + G2_OFF + xcd * 8192u + (unsigned)lane * 128u);
        const unsigned rem0 = (h ? cb >= 4u : rbp < 16u && rbp * 2u - mem * 5u >= 5u) ? 1u << 16 : 0u;
        const unsigned rem1 = (h ? cb >= 4u : rbp < 16u && rbp * 2u + 1u - mem * 5u >= 5u) ? 1u << 17 : 0u;
        const unsigned fv = (h ? FB_OFF + cb * 512u : FA_OFF + rbp * 512u) + s * 128u + (lane & 15) * 8u;
        const unsigned sv0 = src0 + (lane & 1) * 8192u + SI * 1024u;
        const unsigned thr = lane < 16 ? call : 1u;
        unsigned known = 0;
        bool dead = false;
#pragma nounroll
        for (unsigned pass = opaque(0u); pass < 2u; pass = opaque(pass + 1u)) {
            for (unsigned n = pass ? 0u : 15u; n < 16u; ++n) {
                const bool tail = n >= 16u - TN;
                const unsigned m0 = (1u << n) | (tail ? rem0 : 0u), m1 = (1u << n) | (tail ? rem1 : 0u);
                const unsigned pv = tail && lane >= 16 ? sv0 + n * sstep : fv;
                bool go = !pass || (known & m0) != 0u;
                for (unsigned it = 1; !dead && !go; ++it) {
                    const v2u f = __builtin_amdgcn_raw_buffer_load_b64(rs, pv, 0, 17);
                    asm volatile("" ::: "memory");
                    const unsigned b0 = (unsigned)__ballot(f.x >= thr), b1 = (unsigned)__ballot(f.y >= thr);
                    known = b0 & b1 & 0xffffu;
                    go = (b0 & m0) != 0u && (b1 & m1) != 0u;
                    if ((it & 255u) == 0u && (rt() - t0 > DEADLINE_TICKS || CTL[8] != 0u)) { dead = true; break; }
                }
                for (unsigned it = 1; !dead && CTL[12 + s] < 8u * n * pass; ++it) {
                    if ((it & 63u) == 0u && CTL[8] != 0u) dead = true;
                    __builtin_amdgcn_s_sleep(1);
                }
                if (dead) break;
                asm volatile("" ::: "memory");
                const unsigned sa = src0 + n * sstep, lvo = pass ? (unsigned)lane * 16u : 0x80000000u;
                while (true) {
#pragma unroll
                    for (int g = 0; g < 4; ++g) {
                        __builtin_amdgcn_raw_ptr_buffer_load_lds(rs, dst + g * 4096, 16, lvo, sa + g * 4096u, 0, 17);
                        __builtin_amdgcn_raw_ptr_buffer_load_lds(rs, dst + g * 4096, 16, lvo, sa + g * 4096u, 1024, 17);
                        __builtin_amdgcn_raw_ptr_buffer_load_lds(rs, dst + g * 4096, 16, lvo, sa + g * 4096u, 2048, 17);
                        __builtin_amdgcn_raw_ptr_buffer_load_lds(rs, dst + g * 4096, 16, lvo, sa + g * 4096u, 3072, 17);
                    }
                    if ((known >> n) & 1u) { WAIT_VM0(); break; }
                    const u64 f = __hip_atomic_load(fp, __ATOMIC_RELAXED, __HIP_MEMORY_SCOPE_SYSTEM);
                    WAIT_VM0();
                    unsigned c[16];
#pragma unroll
                    for (int g = 0; g < 16; ++g) c[g] = *(const volatile LAS unsigned*)(dst + g * 1024 + lane * 16);
#pragma unroll
                    for (int g = 1; g < 16; ++g) c[0] = min(c[0], c[g]);
                    if (!pass || __all(c[0] != 0u)) break;
                    known = (unsigned)__ballot((unsigned)f >= call && (unsigned)(f >> 32) >= call) & 0xffffu;
                    if (rt() - t0 > DEADLINE_TICKS || CTL[8] != 0u) { dead = true; break; }
                }
                if (dead) break;
                CTL[pass ? s * 2u + h : 19u] = n + 1u;
            }
            if (dead) {
                __hip_atomic_store(tmo, call, __ATOMIC_RELAXED, __HIP_MEMORY_SCOPE_AGENT);
                CTL[8] = 1u; CTL[9] = 1u;
            }
            if (wave == 8) {
                const unsigned want = pass ? call : 0u;
                unsigned best = 0u, sbits = 0x80000000u;
                bool sig = false;
                const unsigned own = 1u << xcd;
                u64 v = __hip_atomic_load(gp, __ATOMIC_RELAXED, __HIP_MEMORY_SCOPE_SYSTEM);
                for (unsigned it = 1;; ++it) {
                    const bool ok = (unsigned)(v >> 32) >= want;
                    const unsigned gm = ok ? (unsigned)v | (pass ^ 1u) : 0u;
                    const bool all = __all(ok), newer = __any(gm > best);
                    if (all && !newer) break;
                    if (!sig && (((unsigned)(__ballot(ok) >> (rank * 8u)) & own) == own)) { sig = true; CTL[pass ? 20u : 19u] = 1u; }
                    if (__builtin_popcountll(__ballot(ok)) < (cb < 4u ? 64 : 48)) __builtin_amdgcn_s_sleep(16);
                    v = __hip_atomic_load(gp, __ATOMIC_RELAXED, __HIP_MEMORY_SCOPE_SYSTEM);
                    if (newer) {
                        best = wave_umax(gm);
                        sbits = best == BAD ? BAD : __float_as_uint(127.0f / __uint_as_float(best)) | (best >= 0x7f800000u ? 0x80000000u : 0u);
                    }
                    if (all) break;
                    if ((it & 63u) == 0u) {
                        if (rt() - t0 > DEADLINE_TICKS ||
                            __builtin_amdgcn_readfirstlane(__hip_atomic_load(tmo, __ATOMIC_RELAXED, __HIP_MEMORY_SCOPE_AGENT)) == call) {
                            __hip_atomic_store(tmo, call, __ATOMIC_RELAXED, __HIP_MEMORY_SCOPE_AGENT);
                            sbits = BAD;
                            break;
                        }
                    }
                }
                CTL[pass ? 16u : 19u] = sbits;
            }
            WAIT_LGKM0();
        }
        return;
    }

    const int wm = wave >> 1, wn = wave & 1;
    const int g = lane >> 4, i = lane & 15;
    const unsigned rb = rbp0 * 2u + ((unsigned)wm >> 1);
    const unsigned sidx = rb < 20u ? rb / 5u : 4u + (rb - 20u) / 3u;
    const unsigned sblk = rb < 20u ? rb % 5u : 5u + (rb - 20u) % 3u;
    const unsigned rloc = ((unsigned)wm & 1u) * 32u;
    const unsigned grow = moved ? rank * 512u + (5u + (mem & 1u) + ((unsigned)wm >> 1)) * 64u + rloc
                                : ((sidx + grp * 4u) & 7u) * 512u + sblk * 64u + rloc;
    const unsigned prow = ((sidx + grp * 4u + 4u) & 7u) * 512u + sblk * 64u + rloc;
    const bool remote = cb >= 4u;
    const bool watch = cb0 < 4u && rbp0 != ((0xfdcau >> (mem * 4u)) & 15u);
    const unsigned ocol = cb * 128u + (unsigned)wn * 64u;
    const unsigned msg = (rbp0 * 4u + (remote ? mem : cb)) * 8u + (unsigned)wave;
    const __amdgpu_buffer_rsrc_t rm = mkrsrc(p.wsr[remote ? (cb - mem) & 7u : 0u], WS_USED);
    const __amdgpu_buffer_rsrc_t ro = mkrsrc(p.out, OUT_BYTES);
    GAS u64* g1 = (GAS u64*)(wsme + G1_OFF + xcd * 1024u + (sl * 8u + (unsigned)wave) * 8u);
    asm volatile("" : "+s"(g1));
    if (lane == 0) __hip_atomic_store(g1 + (G0_OFF - G1_OFF) / 8, (u64)(call - 1u) << 32, __ATOMIC_RELAXED, __HIP_MEMORY_SCOPE_WORKGROUP);
    if (watch) {
        const v4u mark = {0x80808080u, 0x80808080u, 0x80808080u, 0x80808080u};
        st128(mark, rm, lane * 16u, D_OFF + msg * 2048u);
        st128(mark, rm, lane * 16u + 1024u, D_OFF + msg * 2048u);
    }
    f32x4 acc[2][4];
#pragma unroll
    for (int m = 0; m < 2; ++m)
#pragma unroll
        for (int f = 0; f < 4; ++f) acc[m][f] = (f32x4){0.f, 0.f, 0.f, 0.f};
    const unsigned a_off0 = (unsigned)(32 * wm + i) * 128u + (unsigned)((g ^ (i & 7)) * 16);
    const unsigned a_off1 = (unsigned)(32 * wm + i) * 128u + (unsigned)(((4 + g) ^ (i & 7)) * 16);
    const unsigned b_off = 16384u + (unsigned)(i & 1) * 4096u + (unsigned)(g * 16 + ((8 * wn + (i >> 1)) ^ (8 * (i & 1)))) * 16u;
    bool dead = false;
    unsigned mycnt = 0;
    const volatile LAS v2u* fullp = (const volatile LAS v2u*)(L8 + CTL_OFF + (lane < 4 ? lane : 4) * 8);
    WAIT_VM0();
    {
        v4u a1[2], b1[4];
        a1[0] = a1[1] = b1[0] = b1[1] = b1[2] = b1[3] = (v4u){0u, 0u, 0u, 0u};
        v2u f = {0u, 0u};
        const unsigned fa = ctl + (unsigned)(lane < 4 ? lane : 4) * 8u;
#define MM(A, B, mm, f0, f1) for (int f_ = f0; f_ < f1; ++f_) acc[mm][f_] = __builtin_amdgcn_mfma_f32_16x16x32_bf16(__builtin_bit_cast(bf16x8, A), __builtin_bit_cast(bf16x8, B[f_]), acc[mm][f_], 0, 0, 0)
#define RD6(A, B, pa, pb) asm volatile("ds_read_b128 %0, %6\n\tds_read_b128 %2, %7\n\tds_read_b128 %3, %7 offset:1024\n\tds_read_b128 %4, %7 offset:2048\n\t" \
                          "ds_read_b128 %5, %7 offset:3072\n\tds_read_b128 %1, %6 offset:2048" \
                          : "=&v"(A[0]), "=&v"(A[1]), "=&v"(B[0]), "=&v"(B[1]), "=&v"(B[2]), "=&v"(B[3]) : "v"(pa), "v"(pb) : "memory")
#pragma nounroll
        for (unsigned n = 0;; ++n) {
            bool av = min(f.x, f.y) > mycnt;
            unsigned m = (unsigned)__ballot(av) & 31u;
            const bool got = m != 0u;
            if (!got) {
                _Pragma("unroll") MM(a1[0], b1, 0, 0, 4);
                _Pragma("unroll") MM(a1[1], b1, 1, 0, 4);
                if (n == (unsigned)KSTEPS) break;
                while (true) {
                    f = *fullp;
                    av = min(f.x, f.y) > mycnt;
                    m = (unsigned)__ballot(av) & 31u;
                    if (m) break;
                    __builtin_amdgcn_s_sleep(1);
                }
            }
            if (m & 16u) { dead = true; break; }
            unsigned key = av ? mycnt * 4u + (unsigned)lane : 1023u;
            key = min(key, (unsigned)__builtin_amdgcn_update_dpp(0, (int)key, 0xb1, 0xf, 0xf, true));
            key = min(key, (unsigned)__builtin_amdgcn_update_dpp(0, (int)key, 0x4e, 0xf, 0xf, true));
            const unsigned s = (unsigned)__builtin_amdgcn_readfirstlane(key) & 3u;
            const unsigned so = s * SLOT_BYTES;
            v4u a0[2], b0[4];
            RD6(a0, b0, so + a_off0, so + b_off);
            __builtin_amdgcn_sched_barrier(0);
            mycnt += ((unsigned)lane == s) ? 1u : 0u;
            const unsigned pa1 = so + a_off1, pb1 = so + b_off + 8192u, fr = ctl + 48u + s * 4u;
            if (got) {
                _Pragma("unroll") MM(a1[0], b1, 0, 0, 4);
                _Pragma("unroll") MM(a1[1], b1, 1, 0, 4);
            }
            __builtin_amdgcn_sched_barrier(0);
            RD6(a1, b1, pa1, pb1);
            asm volatile("s_waitcnt lgkmcnt(9)" : "+v"(a0[0]), "+v"(b0[0]), "+v"(b0[1]) :: "memory");
            _Pragma("unroll") MM(a0[0], b0, 0, 0, 2);
            __builtin_amdgcn_sched_barrier(0);
            asm volatile("s_waitcnt lgkmcnt(7)" : "+v"(b0[2]), "+v"(b0[3]) :: "memory");
            _Pragma("unroll") MM(a0[0], b0, 0, 2, 4);
            __builtin_amdgcn_sched_barrier(0);
            asm volatile("s_waitcnt lgkmcnt(6)\n\tds_read_b64 %1, %2" : "+v"(a0[1]), "=&v"(f) : "v"(fa) : "memory");
            _Pragma("unroll") MM(a0[1], b0, 1, 0, 4);
            __builtin_amdgcn_sched_barrier(0);
            asm volatile("s_waitcnt lgkmcnt(0)\n\t"
                         "s_mov_b64 exec, 1\n\t"
                         "ds_add_u32 %7, %8\n\t"
                         "s_mov_b64 exec, -1"
                         : "+v"(a1[0]), "+v"(a1[1]), "+v"(b1[0]), "+v"(b1[1]), "+v"(b1[2]), "+v"(b1[3]), "+v"(f)
                         : "v"(fr), "v"(1u) : "memory");
            __builtin_amdgcn_sched_barrier(0);
        }
    }

    u64 gran;
    {
        float mx = 0.f;
#pragma unroll
        for (int m = 0; m < 2; ++m)
#pragma unroll
            for (int f = 0; f < 4; ++f)
#pragma unroll
                for (int r = 0; r < 4; ++r) mx = fmaxf(mx, acc[m][f][r]);
        const unsigned wmx = dead ? BAD : wave_umax(__float_as_uint(mx));
        gran = ((u64)call << 32) | wmx;
        if (lane == 0) __hip_atomic_store(g1 + (G0_OFF - G1_OFF) / 8, gran, __ATOMIC_RELAXED, __HIP_MEMORY_SCOPE_WORKGROUP);
    }
    auto overdue = [&](unsigned it) { if (it == 4096u && lane == 0) __hip_atomic_store(g1, gran, __ATOMIC_RELAXED, __HIP_MEMORY_SCOPE_SYSTEM); };
    asm volatile("" : "+v"(acc[0][0]), "+v"(acc[0][1]), "+v"(acc[0][2]), "+v"(acc[0][3]),
                      "+v"(acc[1][0]), "+v"(acc[1][1]), "+v"(acc[1][2]), "+v"(acc[1][3]) :: "memory");
    const unsigned xb = ((unsigned)wave & 3u) * SLOT_BYTES + ((unsigned)wave >> 2) * 16384u;
    const unsigned ovo = (unsigned)g * 2048u + (unsigned)i * 4u;
    if (remote) {
        unsigned mk = 0;
#pragma unroll
        for (int j = 7; j >= 0; --j)
#pragma unroll
            for (int f = 3; f >= 0; --f) mk = __builtin_amdgcn_alignbit(mk, __float_as_uint(acc[j >> 2][f][j & 3]), 31u);
        mk = ~mk;
        if (mk == 0u) { mk = 1u; acc[0][0][0] = 0.f; }
        unsigned o[8];
        const unsigned cnt = opaque(positions(mk, o));
        for (unsigned it = 1; !dead && CTL[12 + (wave & 3)] < 128u; ++it)
            if ((it & 63u) == 0u && CTL[8] != 0u) dead = true;
        asm volatile("" ::: "memory");
        LAS f32x4* const xp = (LAS f32x4*)(L8 + xb + 16 + lane * 16);
        for (unsigned q = cnt + (unsigned)lane; q < max(1024u, cnt + 16u); q += 64u) *(LAS float*)(L8 + xb + 16 + q * 4u) = 0.f;
#pragma unroll
        for (int j = 0; j < 8; ++j) {
            unsigned a = xb + 16u + o[j] * 4u;
#pragma unroll
            for (int f = 0; f < 4; ++f) {
                const float v = acc[j >> 2][f][j & 3];
                const unsigned sg = (unsigned)(__float_as_int(v) >> 31);
                asm volatile("ds_write_b32 %0, %1 offset:%2" :: "v"(a | (sg & 0x40000u)), "v"(v), "n"(f * 4) : "memory");
                a += sg << 2;
            }
        }
        f32x4 d[8];
#pragma unroll
        for (int c = 0; c < 8; ++c) d[c] = xp[c * 64];
        asm volatile("s_waitcnt lgkmcnt(0)" : "+v"(d[0]), "+v"(d[1]), "+v"(d[2]), "+v"(d[3]),
                                                       "+v"(d[4]), "+v"(d[5]), "+v"(d[6]), "+v"(d[7]) :: "memory");
        const unsigned lim = (unsigned)(lane & ~3) * 4u;
        const unsigned dso = opaque(D_OFF + msg * 2048u);
        unsigned sb;
        for (unsigned it = 1; __builtin_amdgcn_readfirstlane(CTL[20] | CTL[16]) == 0u; ++it) overdue(it);
        __builtin_amdgcn_raw_buffer_store_b32(mk, rm, lane * 4u, M_OFF + msg * 256u, 17);
        do { sb = __builtin_amdgcn_readfirstlane(CTL[16]); } while (sb == 0u);
        sb = opaque(sb);
        if (sb != BAD) {
            const float inv = __uint_as_float(sb);
            v4u q;
#pragma unroll
            for (int c = 0; c < 4; ++c) q[c] = quant4(d[c], inv, 12582912.f);
            asm volatile("" : "+v"(q) :: "memory");
            int acked = 0;
            while (acked < wave + 1 - OUT_WIN) acked = (int)__builtin_amdgcn_readfirstlane(CTL[17]);
            st128(q, rm, lane * 16u, dso);
#pragma unroll
            for (int c = 4; c < 8; ++c) {
                if (cnt <= 256u * c) break;
                const unsigned e = quant4(d[c], inv, 12582912.f);
                if (lim + 256u * c < cnt) __builtin_amdgcn_raw_buffer_store_b32(e, rm, lane * 4u + 256u * c, dso, 17);
            }
        }
        asm volatile("s_waitcnt vmcnt(0)\n\t"
                     "s_mov_b64 exec, 1\n\t"
                     "ds_add_u32 %0, %1 offset:68\n\t"
                     "s_mov_b64 exec, -1" :: "v"(ctl), "v"(1u) : "memory");
        return;
    }

#pragma unroll
    for (int m = 0; m < 2; ++m)
#pragma unroll
        for (int f = 0; f < 4; ++f)
#pragma unroll
            for (int r = 0; r < 4; ++r) acc[m][f][r] = __int_as_float(max(__float_as_int(acc[m][f][r]), 0));
    asm volatile("" : "+v"(acc[0][0]), "+v"(acc[0][1]), "+v"(acc[0][2]), "+v"(acc[0][3]),
                      "+v"(acc[1][0]), "+v"(acc[1][1]), "+v"(acc[1][2]), "+v"(acc[1][3]) :: "memory");
    unsigned sb;
    for (unsigned it = 1; (sb = __builtin_amdgcn_readfirstlane(CTL[16])) == 0u; ++it) overdue(it);
    dead = sb == BAD;
    unsigned mk = __builtin_amdgcn_raw_buffer_load_b32(rm, lane * 4u, M_OFF + msg * 256u, 17);
    {
        const float inv = dead ? 0.f : __uint_as_float(sb);
        const float magic = dead ? 12583040.f : 12582912.f;
#pragma unroll
        for (int m = 0; m < 2; ++m) {
            const f32x2 iv = {inv, inv}, mg = {magic, magic};
            f32x2 z[4][2];
#pragma unroll
            for (int f = 0; f < 4; ++f) {
                z[f][0] = __builtin_elementwise_fma((f32x2){acc[m][f][0], acc[m][f][1]}, iv, mg);
                z[f][1] = __builtin_elementwise_fma((f32x2){acc[m][f][2], acc[m][f][3]}, iv, mg);
            }
#pragma unroll
            for (int r = 0; r < 4; ++r) {
                const unsigned lo = __builtin_amdgcn_perm(__float_as_uint(z[1][r >> 1][r & 1]), __float_as_uint(z[0][r >> 1][r & 1]), 0x0c0c0400u);
                const unsigned hi = __builtin_amdgcn_perm(__float_as_uint(z[3][r >> 1][r & 1]), __float_as_uint(z[2][r >> 1][r & 1]), 0x04000c0cu);
                __builtin_amdgcn_raw_buffer_store_b32(lo | hi, ro, ovo + r * 512u, (grow + 16u * m) * 512u + ocol, 17);
            }
        }
    }
    if (watch && !dead) {
        unsigned ra[8], sel[8], o[8];
        bool late = false;
        for (unsigned it = 1; __any(mk == 0u); ++it) {
            mk = __builtin_amdgcn_raw_buffer_load_b32(rm, lane * 4u, M_OFF + msg * 256u, 17);
            if ((it & 63u) == 0u && (rt() - t0 > DEADLINE_TICKS ||
                __builtin_amdgcn_readfirstlane(__hip_atomic_load(tmo, __ATOMIC_RELAXED, __HIP_MEMORY_SCOPE_AGENT)) == call)) { late = true; break; }
        }
        __builtin_amdgcn_raw_buffer_store_b32(0u, rm, lane * 4u, M_OFF + msg * 256u, 17);
        const unsigned cnt = positions(mk, o);
#pragma unroll
        for (int j = 0; j < 8; ++j) {
            const unsigned ps = (((mk >> (4 * j)) & 15u) * 0x00204081u) & 0x01010101u;
            unsigned rk = (ps << 8) | (o[j] & 3u);
            rk += rk << 8;
            asm("" : "+v"(rk));
            rk += rk << 16;
            const unsigned am = ps + 0x0f0f0f0fu;
            sel[j] = (rk & ~am) | (0x0c0c0c0cu & am);
            ra[j] = xb + (o[j] & ~3u);
        }
        unsigned vo1 = (unsigned)lane * 16u + 1024u < cnt ? (unsigned)lane * 16u + 1024u : 0x80000000u;
        asm volatile("" : "+v"(ra[0]), "+v"(ra[1]), "+v"(ra[2]), "+v"(ra[3]), "+v"(ra[4]), "+v"(ra[5]), "+v"(ra[6]), "+v"(ra[7]),
                          "+v"(sel[0]), "+v"(sel[1]), "+v"(sel[2]), "+v"(sel[3]), "+v"(sel[4]), "+v"(sel[5]), "+v"(sel[6]), "+v"(sel[7]), "+v"(vo1) :: "memory");
        const unsigned pso = opaque(prow * 512u + ocol);
        while (!late) {
            unsigned got, t, t2;
#define DLD(a, b) "buffer_load_dwordx4 v[" #a "], %3, %5, %6 offen sc0 sc1\n\tbuffer_load_dwordx4 v[" #b "], %4, %5, %6 offen sc0 sc1\n\ts_sleep %9\n\t"
#define DCK(k, r0, r1, r2, r3, r4, r5, r6, r7) "s_waitcnt vmcnt(%10)\n\t" \
            "v_or3_b32 %1, v" #r0 ", v" #r1 ", v" #r2 "\n\tv_or3_b32 %2, v" #r3 ", v" #r4 ", v" #r5 "\n\tv_or3_b32 %1, %1, v" #r6 ", v" #r7 "\n\t" \
            "v_or_b32 %1, %1, %2\n\tv_and_b32 %1, 0x80808080, %1\n\tv_cmp_ne_u32 vcc, 0, %1\n\ts_cbranch_vccz .Ldg" #k "%=\n\t" DLD(r0:r3, r4:r7)
#define DWR(k, r0, r1, r2, r3, r4, r7) ".Ldg" #k "%=:\n\tds_write2st64_b32 %7, v" #r0 ", v" #r1 " offset1:1\n\tds_write2st64_b32 %7, v" #r2 ", v" #r3 " offset0:2 offset1:3\n\t" \
            "ds_write_b128 %8, v[" #r4 ":" #r7 "] offset:1024\n\ts_mov_b32 %0, 1\n\t"
            asm volatile(DLD(104:107, 108:111)
                         "s_movk_i32 %0, 100\n"
                         ".Ldl%=:\n\t"
                         DCK(0, 104, 105, 106, 107, 108, 109, 110, 111)
                         "s_sub_u32 %0, %0, 1\n\t"
                         "s_cbranch_scc0 .Ldl%=\n\t"
                         "s_mov_b32 %0, 0\n\t"
                         "s_branch .Lde%=\n"
                         DWR(0, 104, 105, 106, 107, 108, 111)
                         "\n.Lde%=:"
                         : "=&s"(got), "=&v"(t), "=&v"(t2)
                         : "v"(lane * 16u), "v"(vo1), "s"(rm), "s"(D_OFF + msg * 2048u), "v"(xb + lane * 4u), "v"(xb + lane * 16u), "n"(0), "n"(0)
                         : "vcc", "scc", "memory", "v104", "v105", "v106", "v107", "v108", "v109", "v110", "v111", "v112", "v113", "v114", "v115",
                           "v116", "v117", "v118", "v119", "v120", "v121", "v122", "v123", "v124", "v125", "v126", "v127");
            if (got) break;
            if (rt() - t0 > DEADLINE_TICKS ||
                __builtin_amdgcn_readfirstlane(__hip_atomic_load(tmo, __ATOMIC_RELAXED, __HIP_MEMORY_SCOPE_AGENT)) == call) late = true;
        }
        if (late) {
            __hip_atomic_store(tmo, call, __ATOMIC_RELAXED, __HIP_MEMORY_SCOPE_AGENT);
            __builtin_amdgcn_raw_buffer_store_b32(0x80808080u, ro, ovo, prow * 512u + ocol, 17);
        } else {
            v2u e[8];
#pragma unroll
            for (int j = 0; j < 8; ++j) {
                const LAS unsigned* rp = (const LAS unsigned*)(L8 + ra[j]);
                e[j] = (v2u){rp[0], rp[1]};
            }
            __builtin_amdgcn_sched_barrier(0);
#pragma unroll
            for (int j = 0; j < 8; ++j)
                __builtin_amdgcn_raw_buffer_store_b32(__builtin_amdgcn_perm(e[j].y, e[j].x, sel[j]), ro, ovo + (j & 3) * 512u, pso + (j >> 2) * 8192u, 17);
        }
    }
}

extern "C" void kernel_launch(int rank, int nranks, ncclComm_t comm, hipStream_t stream,
                              void* const* d_in, const int* in_sizes, int n_in,
                              void* d_out, int out_size, const int* in_dtypes, int out_dtype,
                              void* const* d_out_all, void* const* d_ws_all, size_t ws_bytes) {
    if (nranks != NR || n_in < 2 || ws_bytes < (size_t)WS_USED + 4096u || (unsigned)rank >= (unsigned)NR ||
        in_sizes[0] != 512 * 4096 || in_sizes[1] != 4096 * 512 || out_size != 4096 * 512) {
        fprintf(stderr, "unexpected problem configuration\n");
        return;
    }
    Params p;
    p.x = d_in[0];
    p.w = d_in[1];
    for (int d = 0; d < NR; ++d) p.wsr[d] = (char*)d_ws_all[(rank + d) & 7];
    p.out = d_out;
    p.rank = rank;
    ag_gemm_kernel<<<dim3(NWG), dim3(1024), 0, stream>>>(p);
}
```
